```python
import numpy as np

kernel_name = "gpu_gemm_rs_m4096_k4096_n4096_bf16_relu"


ABSMAX_RTOL = 0.008

COMPUTE_DTYPE = "bfloat16"
EPILOGUE = "relu"
IN_QUANT = None
OUT_QUANT = None
IN_DTYPES = {"x": "bfloat16", "w_mat": "bfloat16"}
WORLD_SIZE = 8

def _to_bf16(x):
    u = np.ascontiguousarray(x, dtype=np.float32).view(np.uint32)
    return ((u + 0x7FFF + ((u >> 16) & 1)) & 0xFFFF0000).view(np.float32)

def setup_inputs():

    x = _to_bf16(np.random.randn(4096, 4096).astype(np.float32))
    w_mat = _to_bf16((np.random.randn(4096, 4096) / np.sqrt(4096)).astype(np.float32))
    return {"x": x, "w_mat": w_mat}

def _epilogue(y):
    return np.maximum(y, 0.0).astype(np.float32)

def reference(x, w_mat):
    y = (x @ w_mat).astype(np.float32)
    return _epilogue(y)
```

```cpp
#include <hip/hip_runtime.h>
#include <rccl/rccl.h>
#include <cstdio>
#include <cstdint>

#define HIP_CHECK(expr) do { hipError_t _e = (expr); if (_e != hipSuccess) { \
    fprintf(stderr, "HIP error %s at %s:%d\n", hipGetErrorString(_e), __FILE__, __LINE__); \
    return; } } while (0)

namespace {

typedef unsigned int u32;
typedef unsigned long long u64;
typedef u32 u32x4 __attribute__((ext_vector_type(4)));
typedef u32 u32x2 __attribute__((ext_vector_type(2)));
typedef float f32x4 __attribute__((ext_vector_type(4)));
typedef float f32x2 __attribute__((ext_vector_type(2)));
typedef short s16x4 __attribute__((ext_vector_type(4)));
typedef short s16x8 __attribute__((ext_vector_type(8)));
typedef __bf16 bf16x8 __attribute__((ext_vector_type(8)));
typedef __bf16 bf16x2 __attribute__((ext_vector_type(2)));
#define AS3 __attribute__((address_space(3)))

#define CFG_NBC 12
#define CFG_NPUSH 16
#define CFG_NRED 16

#define CFG_GATE 46
#define CFG_PW 4
#define CFG_CW 4
#define CFG_NPS (CFG_NPUSH * CFG_PW / 4)
#define CFG_RW 4
#define CFG_DLY_MUL 55u
#define CFG_DLY_SUB 200u
#define CFG_OUT_AUX 0
#define CFG_STG0 50u
#define CFG_STG 7u
constexpr int NRANK = 8;
constexpr int PW = CFG_PW, RW = CFG_RW, CW = CFG_CW;
constexpr int NBC = CFG_NBC;
constexpr int NBI = 32 - NBC;
constexpr int RC = 16 * NBC;
constexpr int NWG = 256;
constexpr int NPUSH = CFG_NPUSH;
constexpr int NRED = CFG_NRED;
constexpr int NGX = (NWG - NPUSH - NRED) / 8;
static_assert((NPUSH + NRED) % 8 == 0, "roles must keep the XCD round-robin aligned");
constexpr int NUI = NBI * 32, NUC = NBC * 32;

constexpr u32 OFF_CTR = 0;
constexpr u32 OFF_TMO = 8192;
constexpr u32 OFF_IF = 16384;
constexpr u32 OFF_YF = 32768;
constexpr u32 OFF_DBG = 65536;
constexpr u32 OFF_Y = 1u << 20;
constexpr u32 OFF_XR = 33u << 20;
constexpr u32 OFF_WR = 37u << 20;
constexpr u32 WS_USED = 45u << 20;
constexpr u64 DEADLINE = 50000000ull;
constexpr int AUX_SYS = 17;
constexpr u32 SRD3 = 0x00020000;

struct Params {
  const char* x;
  const char* w;
  char* out;
  char* ws[NRANK];
  int rank;
  int pad;
  union { unsigned char send[48]; u32 send_w[12]; };
  union { unsigned short gt[8][64]; u32 gt_w[256]; };
  union { unsigned short ru[1024]; u32 ru_w[512]; };
};
__device__ __forceinline__ u32 tab8(const u32* t, u32 i) { return (t[i >> 2] >> ((i & 3u) * 8u)) & 0xffu; }
__device__ __forceinline__ u32 tab16(const u32* t, u32 i) { return (t[i >> 1] >> ((i & 1u) * 16u)) & 0xffffu; }

#define WAIT_VM0() do { asm volatile("" ::: "memory"); __builtin_amdgcn_s_waitcnt(0x0F70); asm volatile("" ::: "memory"); } while (0)
#define WAIT_LGKM0() do { asm volatile("" ::: "memory"); __builtin_amdgcn_s_waitcnt(0xC07F); asm volatile("" ::: "memory"); } while (0)
__device__ __forceinline__ u32 rfl(u32 v) { return __builtin_amdgcn_readfirstlane(v); }
__device__ __forceinline__ __amdgpu_buffer_rsrc_t mkrs(const void* p, u32 bytes) {
  return __builtin_amdgcn_make_buffer_rsrc((void*)p, (short)0, (int)bytes, SRD3);
}
__device__ __forceinline__ u32 vrow0(u32 grp, u32 vb) {
  const u32 h = vb >> 2, o = vb & 3;
  return h < (u32)NBI ? 512u * (4u * grp + o) + RC + 16u * h : 512u * (4u * (grp ^ 1u) + o) + 16u * (h - NBI);
}

#define STAMPC(idx) do {} while (0)
#define STAMP(idx) do {} while (0)
#define STAMPW(idx) do {} while (0)

__device__ __forceinline__ bool wait_words(const u32* word, u32* tmo, u32 seq) {
  const u64 t0 = __builtin_amdgcn_s_memrealtime();
  for (u32 it = 0;; ++it) {
    const u32 v = __hip_atomic_load(word, __ATOMIC_RELAXED, __HIP_MEMORY_SCOPE_SYSTEM);
    if (__builtin_amdgcn_ballot_w64(v < seq) == 0) return true;
    if ((it & 7u) == 7u) {
      const u32 t = __hip_atomic_load(tmo, __ATOMIC_RELAXED, __HIP_MEMORY_SCOPE_AGENT);
      if (rfl(t) == seq || __builtin_amdgcn_s_memrealtime() - t0 > DEADLINE) {
        __hip_atomic_store(tmo, seq, __ATOMIC_RELAXED, __HIP_MEMORY_SCOPE_AGENT);
        return false;
      }
    }
    __builtin_amdgcn_s_sleep(2);
  }
}

#define CFG_GLDS_BITS "sc0 sc1"
template <u32 IMM>
__device__ __forceinline__ void glds(u32 m0v, u32 voff, __amdgpu_buffer_rsrc_t rs, u32 soff) {
  u32 keep;
  asm volatile("s_mov_b32 %0, m0\n\ts_add_u32 m0, %1, %5\n\ts_nop 2\n\t"
               "buffer_load_dwordx4 %2, %3, %4 offen " CFG_GLDS_BITS " lds\n\ts_mov_b32 m0, %0"
               : "=&s"(keep) : "s"(m0v), "v"(voff), "s"(rs), "s"(soff), "i"(IMM) : "memory", "scc");
}

__device__ __forceinline__ float bflo(u32 v) { return __builtin_bit_cast(float, v << 16); }
__device__ __forceinline__ float bfhi(u32 v) { return __builtin_bit_cast(float, v & 0xffff0000u); }
__device__ __forceinline__ u32 pkbf(float a, float b) {
  f32x2 f = {a, b};
  return __builtin_bit_cast(u32, __builtin_convertvector(f, bf16x2));
}

__device__ __forceinline__ void reduce_units(const Params& p, u32 rank, char* me, u32 seq, u32 type, u32 ubase,
                                             u32 count, u32 first, u32 step, u32 lane) {
  const u32 grp = rank >> 2, q = rank & 3;
  u32* tmo = (u32*)(me + OFF_TMO);
  const u32 pg = (grp ^ type) * 4u;
  __amdgpu_buffer_rsrc_t rp0 = mkrs(p.ws[(pg + ((q + 1) & 3)) & 7], WS_USED);
  __amdgpu_buffer_rsrc_t rp1 = mkrs(p.ws[(pg + ((q + 2) & 3)) & 7], WS_USED);
  __amdgpu_buffer_rsrc_t rp2 = mkrs(p.ws[(pg + ((q + 3) & 3)) & 7], WS_USED);
  __amdgpu_buffer_rsrc_t rp3 = mkrs(p.ws[(pg + q) & 7], WS_USED);
  __amdgpu_buffer_rsrc_t rme = mkrs(me, WS_USED);
  __amdgpu_buffer_rsrc_t ro = mkrs(p.out, 512u * 8192u);
  const u32 ybase = OFF_Y + (seq & 1u) * (16u << 20) + q * 4096u;
  const u32 g = lane >> 4;
  const u32 vo = (lane & 15u) * 8192u + (16u * (g & 1u) + 8u * (g >> 1)) * 2u;
  const u32 vl = lane * 16u;
  if (first >= count) return;
  u32 e = tab16(p.ru_w, ubase + first);
  u32x4 fl = __builtin_amdgcn_raw_buffer_load_b128(rme, 0, OFF_YF + (type * 512u + (e >> 6) * 32u + (e & 31u)) * 16u, AUX_SYS);
  for (u32 u = first; u < count; u += step) {
    const u32 h = e >> 5, b = e & 31u;
    const u32 fmin = min(min(rfl(fl.x), rfl(fl.y)), min(rfl(fl.z), rfl(fl.w)));
    bool ok = true;
    if (fmin < seq)
      ok = wait_words((const u32*)(me + OFF_YF + (type * 512u + (h >> 1) * 32u + b) * 16u) + (lane & 3u), tmo, seq);
    const u32 un = u + step;
    const u32 en = tab16(p.ru_w, ubase + (un < count ? un : u));
    fl = __builtin_amdgcn_raw_buffer_load_b128(rme, 0, OFF_YF + (type * 512u + (en >> 6) * 32u + (en & 31u)) * 16u, AUX_SYS);
    const u32 so = ybase + e * 16384u;
    const u32 orow = type == 0 ? RC + 16u * h : 16u * (h - NBI);
    const u32 oo = orow * 8192u + b * 256u;
    if (ok) {
      u32x4 d[4][4];
#pragma unroll
      for (int i = 0; i < 4; ++i) {
        d[0][i] = __builtin_amdgcn_raw_buffer_load_b128(rp0, vl, so + i * 1024u, AUX_SYS);
        d[1][i] = __builtin_amdgcn_raw_buffer_load_b128(rp1, vl, so + i * 1024u, AUX_SYS);
        d[2][i] = __builtin_amdgcn_raw_buffer_load_b128(rp2, vl, so + i * 1024u, AUX_SYS);
        d[3][i] = __builtin_amdgcn_raw_buffer_load_b128(rp3, vl, so + i * 1024u, AUX_SYS);
      }
#pragma unroll
      for (int i = 0; i < 4; ++i) {
        u32x4 r;
#pragma unroll
        for (int c = 0; c < 4; ++c) {
          const float lo = (bflo(d[0][i][c]) + bflo(d[1][i][c])) + (bflo(d[2][i][c]) + bflo(d[3][i][c]));
          const float hi = (bfhi(d[0][i][c]) + bfhi(d[1][i][c])) + (bfhi(d[2][i][c]) + bfhi(d[3][i][c]));
          r[c] = pkbf(fmaxf(lo, 0.f), fmaxf(hi, 0.f));
        }
        __builtin_amdgcn_raw_buffer_store_b128(r, ro, vo + oo + i * 64u, 0, CFG_OUT_AUX);
      }
    } else {
      const u32x4 nanv = {0x7fc07fc0u, 0x7fc07fc0u, 0x7fc07fc0u, 0x7fc07fc0u};
#pragma unroll
      for (int i = 0; i < 4; ++i) __builtin_amdgcn_raw_buffer_store_b128(nanv, ro, vo + oo + i * 64u, 0, 0);
    }
    e = en;
  }
}

struct Frag { bf16x8 x[4]; bf16x8 w[4]; };

template <int STAGE, int KS>
__device__ __forceinline__ void read_frag(Frag& f, const char AS3* lds, u32 xa0, u32 xa1, const u32 (&wa)[4]) {
  const u32 xa = KS ? xa1 : xa0;
  f.x[0] = *(const bf16x8 AS3*)(lds + xa + STAGE * 32768);
#pragma unroll
  for (int nt = 0; nt < 4; ++nt) {
    const s16x4 lo = __builtin_amdgcn_ds_read_tr16_b64_v4i16((s16x4 AS3*)(lds + wa[nt] + STAGE * 32768 + KS * 8192));
    const s16x4 hi = __builtin_amdgcn_ds_read_tr16_b64_v4i16((s16x4 AS3*)(lds + wa[nt] + STAGE * 32768 + KS * 8192 + 1024));
    const s16x8 v = __builtin_shufflevector(lo, hi, 0, 1, 2, 3, 4, 5, 6, 7);
    f.w[nt] = __builtin_bit_cast(bf16x8, v);
  }
#pragma unroll
  for (int mt = 1; mt < 4; ++mt)
    f.x[mt] = *(const bf16x8 AS3*)(lds + xa + STAGE * 32768 + mt * 2048);
}

#define SCHED() __builtin_amdgcn_sched_barrier(0)
#define BAR() __builtin_amdgcn_s_barrier()
constexpr u32 LDS_SEQ = 131072, LDS_FAIL = 131076;

__global__ void __launch_bounds__(512) fused_gemm_rs(const Params p) {
  __shared__ __attribute__((aligned(1024))) char lds_raw[131072 + 64];
  const u32 tid = threadIdx.x, lane = tid & 63u;
  const u32 wave = rfl(tid >> 6);
  const u32 rank = (u32)p.rank & 7u, grp = rank >> 2, q = rank & 3u;
  char* me = p.ws[rank];
  u32* tmo = (u32*)(me + OFF_TMO);
  u32* ctr = (u32*)(me + OFF_CTR) + blockIdx.x * 8u + wave;
  const u32 bid = blockIdx.x;
  STAMP(0);

  if (bid < (u32)(NPUSH + NRED)) {
    const bool cross = bid < (u32)NPUSH;
    if (wave >= (cross ? (u32)(PW > CW ? PW : CW) : (u32)RW)) return;
    u32 seq;
    if (cross) {
      constexpr u32 NPS = CFG_NPS;
      const u32 pw = bid * PW + wave, dq = pw & 3u, ps0 = pw >> 2;
      const u32 ps = (wave < (u32)PW && ps0 < NPS) ? ps0 : 384u;
      const u32 dst = ((grp ^ 1u) * 4u + dq) & 7u;
      __amdgpu_buffer_rsrc_t rd = mkrs(p.ws[dst], WS_USED);
      const u32 lr = lane >> 4, lc = (lane & 15u) * 16u;
      const u32 vx = lr * 1024u + lc, vw = lr * 8192u + lc;
      u32x4 v[4];
      u32 dso = 0, dvo = 0, dstep = 0, dfl = 0;
      auto prep = [&](u32 gi, u32x4 (&r)[4], u32& o_so, u32& o_vo, u32& o_step, u32& o_fl) {
        const u32 c = tab8(p.send_w, gi >> 3), part = gi & 7u;
        const bool isx = c < 16u;
        const u32 vb = c * 8u + part, b = c - 16u;
        const u32 sso = isx ? vrow0(grp ^ 1u, vb) * 1024u + dq * 256u : (128u * dq + 16u * part) * 8192u + b * 256u;
        o_so = isx ? OFF_XR + vb * 16384u + q * 256u : OFF_WR + (128u * q + 16u * part) * 8192u + b * 256u;
        o_vo = isx ? vx : vw;
        o_step = isx ? 4096u : 32768u;
        o_fl = OFF_IF + c * 256u + (q * 8u + part) * 4u;
        __amdgpu_buffer_rsrc_t rs = mkrs(isx ? p.x : p.w, 4u << 20);
#pragma unroll
        for (int i = 0; i < 4; ++i) r[i] = __builtin_amdgcn_raw_buffer_load_b128(rs, o_vo, sso + i * o_step, 0);
      };
      const u64 tstart = __builtin_amdgcn_s_memrealtime();
      prep(ps < 384u ? ps : 0u, v, dso, dvo, dstep, dfl);
      asm volatile("" ::: "memory");
      seq = rfl(__hip_atomic_load(ctr, __ATOMIC_RELAXED, __HIP_MEMORY_SCOPE_SYSTEM) + 1u);
      WAIT_VM0();
      while (__builtin_amdgcn_s_memrealtime() - tstart < (u64)(CFG_STG0 + ps0 * CFG_STG)) __builtin_amdgcn_s_sleep(1);
      const u32 par = seq & 1u;
      const u32 fvo = lane == 0 ? 0u : 0x80000000u;
      for (u32 gi = ps; gi < 384u; gi += NPS) {
        u32x4 vn[4];
        u32 nso, nvo, nstep, nfl;
        const u32 pso = dso + par * (dstep == 4096u ? (2u << 20) : (4u << 20));
#pragma unroll
        for (int i = 0; i < 4; ++i) __builtin_amdgcn_raw_buffer_store_b128(v[i], rd, dvo + pso + i * dstep, 0, AUX_SYS);
        asm volatile("" ::: "memory");
        const u32 gn = gi + NPS < 384u ? gi + NPS : gi;
        prep(gn, vn, nso, nvo, nstep, nfl);
        WAIT_VM0();
        __builtin_amdgcn_raw_buffer_store_b32(seq, rd, fvo, dfl, AUX_SYS);
#pragma unroll
        for (int i = 0; i < 4; ++i) v[i] = vn[i];
        dso = nso; dvo = nvo; dstep = nstep; dfl = nfl;
      }
      STAMPW(1 + (wave & 3));
      const u32 c = tab8(p.send_w, CFG_GATE + (lane >> 5));
      (void)wait_words((const u32*)(me + OFF_IF + c * 256u) + (lane & 31u), tmo, seq);
    } else {
      seq = rfl(__hip_atomic_load(ctr, __ATOMIC_RELAXED, __HIP_MEMORY_SCOPE_SYSTEM) + 1u);
    }
    __hip_atomic_store(ctr, seq, __ATOMIC_RELAXED, __HIP_MEMORY_SCOPE_SYSTEM);
    reduce_units(p, rank, me, seq, cross ? 1u : 0u, cross ? NUI : 0u, cross ? NUC : NUI,
                 cross ? (wave < (u32)CW ? bid * CW + wave : 4096u) : (bid - NPUSH) * RW + wave, cross ? NPUSH * CW : NRED * RW, lane);
    STAMPW(5 + (wave & 3));
    return;
  }

  const char AS3* lds = (const char AS3*)lds_raw;
  const u32 gj = bid - (NPUSH + NRED), xcd = gj & 7u, slot = gj >> 3;
  const u32 ntile = (64u - slot + NGX - 1u) / NGX;
  const u32 g = lane >> 4, l15 = lane & 15u;
  __amdgpu_buffer_rsrc_t rme = mkrs(me, WS_USED);

  if (wave >= 4u) {
    const u64 tstart = __builtin_amdgcn_s_memrealtime();
    __builtin_amdgcn_s_setprio(2);
    const u32 lw = wave - 4u;
    u32 vX[2], vW[2];
#pragma unroll
    for (u32 j = 0; j < 2; ++j) {
      vX[j] = (8u * j + (lane >> 3)) * 1024u + (((lane & 7u) ^ (4u * j + g)) * 16u);
      vW[j] = g * 8192u + ((((l15 >> 1) ^ (g | (j << 2))) * 2u + (l15 & 1u)) * 16u);
    }
    const u32 m0b = (u32)(uintptr_t)lds + lw * 4096u;
    const u32 rot = (slot * 3u) & 7u;
    const __amdgpu_buffer_rsrc_t rxo = mkrs(p.x, 4u << 20), rwo = mkrs(p.w, 4u << 20);
    const u32 total = ntile * 16u;
    u32 issued = 0, seq = 0;
    u32 sx0 = 0, sx1 = 0, sw = 0, ea = 0, eb = 0;
    bool rem = false;
    {
      const u32 av = tab16(p.gt_w, xcd * 64u + slot) >> 9;
      const u32 dl = av * CFG_DLY_MUL;
      if (dl > CFG_DLY_SUB)
        while (__builtin_amdgcn_s_memrealtime() - tstart < (u64)(dl - CFG_DLY_SUB)) __builtin_amdgcn_s_sleep(4);
    }
    for (int n = -1; n < (int)total; ++n) {
      while ((int)issued <= n + 3 && issued < total) {
        const u32 st = issued & 15u;
        if (st == 0u) {
          const u32 e = tab16(p.gt_w, xcd * 64u + slot + (issued >> 4) * NGX);
          ea = (e >> 5) & 15u; eb = e & 31u;
          const u32 vb = ea * 8u + lw * 2u;
          sx0 = vrow0(grp, vb) * 1024u; sx1 = vrow0(grp, vb + 1u) * 1024u;
          sw = lw * 16u * 8192u + eb * 256u;
          rem = false;
        } else if (st == 8u) {
          seq = rfl(*(volatile const u32 AS3*)(lds + LDS_SEQ));
          const u32* fw = (const u32*)(me + OFF_IF + (lane < 32u ? ea : 16u + eb) * 256u) + (lane & 31u);
          if ((int)issued <= n + 1) {
            if (!wait_words(fw, tmo, seq)) *(volatile u32 AS3*)(lds + LDS_FAIL) = seq;
          } else {
            const u32 v = __hip_atomic_load(fw, __ATOMIC_RELAXED, __HIP_MEMORY_SCOPE_SYSTEM);
            if (__builtin_amdgcn_ballot_w64(v < seq) != 0) break;
          }
          const u32 par = seq & 1u;
          sx0 = OFF_XR + par * (2u << 20) + (ea * 8u + lw * 2u) * 16384u; sx1 = sx0 + 16384u;
          sw = OFF_WR + par * (4u << 20) + lw * 16u * 8192u + eb * 256u;
          rem = true;
        }
        const __amdgpu_buffer_rsrc_t rx = rem ? rme : rxo, rw = rem ? rme : rwo;
        const u32 kk = (issued + rot) & 7u;
        const u32 m0s = m0b + (issued & 3u) * 32768u;
        const u32 kx = kk * 128u, kw = sw + kk * (64u * 8192u);
        glds<0>(m0s, vX[0], rx, sx0 + kx);
        glds<1024>(m0s, vX[1], rx, sx0 + kx);
        glds<16384>(m0s, vW[0], rw, kw);
        glds<16384 + 1024>(m0s, vW[0], rw, kw + 32768u);
        glds<2048>(m0s, vX[0], rx, sx1 + kx);
        glds<3072>(m0s, vX[1], rx, sx1 + kx);
        glds<16384 + 2048>(m0s, vW[1], rw, kw + 65536u);
        glds<16384 + 3072>(m0s, vW[1], rw, kw + 98304u);
        ++issued;
        if (n < 0 && issued == 2u) break;
      }
      const int ahead = (int)issued - (n + 2);
      if (ahead >= 2) asm volatile("s_waitcnt vmcnt(16) lgkmcnt(0)" ::: "memory");
      else if (ahead == 1) asm volatile("s_waitcnt vmcnt(8) lgkmcnt(0)" ::: "memory");
      else asm volatile("s_waitcnt vmcnt(0) lgkmcnt(0)" ::: "memory");
      BAR();
      if ((n & 15) == 15 && n >= 0) BAR();
    }
    return;
  }

  const u32 seq = rfl(__hip_atomic_load(ctr, __ATOMIC_RELAXED, __HIP_MEMORY_SCOPE_SYSTEM) + 1u);
  __hip_atomic_store(ctr, seq, __ATOMIC_RELAXED, __HIP_MEMORY_SCOPE_SYSTEM);
  if (wave == 0) {
    *(volatile u32 AS3*)(lds + LDS_SEQ) = seq;
    *(volatile u32 AS3*)(lds + LDS_FAIL) = 0u;
  }
  const u32 par = seq & 1u;
  const u32 wm = wave >> 1, wn = wave & 1u;
  const u32 xa0 = (64u * wm + l15) * 128u + ((g ^ (l15 >> 1)) * 16u);
  const u32 xa1 = xa0 ^ 64u;
  u32 wa[4];
  {
    const u32 qq = l15 >> 2, pp = lane & 3u, f = qq | ((g & 1u) << 2);
#pragma unroll
    for (u32 nt = 0; nt < 4; ++nt) wa[nt] = 16384u + (8u * g + qq) * 256u + (((4u * wn + nt) ^ f) * 32u) + 8u * pp;
  }
  asm volatile("s_waitcnt lgkmcnt(0)" ::: "memory");
  BAR();
  Frag f0, f1;
  read_frag<0, 0>(f0, lds, xa0, xa1, wa);

#define MMA16(F) \
  _Pragma("unroll") for (int mt = 0; mt < 4; ++mt) \
    _Pragma("unroll") for (int nt = 0; nt < 4; ++nt) \
      acc[mt][nt] = __builtin_amdgcn_mfma_f32_16x16x32_bf16(F.w[nt], F.x[mt], acc[mt][nt], 0, 0, 0);
#define INTERLEAVE() \
  _Pragma("unroll") for (int i = 0; i < 6; ++i) { \
    __builtin_amdgcn_sched_group_barrier(0x008, 1, 0); \
    __builtin_amdgcn_sched_group_barrier(0x100, 2, 0); \
  } \
  __builtin_amdgcn_sched_group_barrier(0x008, 10, 0);
#define STEP(S) \
  SCHED(); \
  read_frag<(S), 1>(f1, lds, xa0, xa1, wa); \
  MMA16(f0) \
  INTERLEAVE() \
  SCHED(); \
  WAIT_LGKM0(); \
  BAR(); \
  SCHED(); \
  read_frag<((S) + 1) & 3, 0>(f0, lds, xa0, xa1, wa); \
  MMA16(f1) \
  INTERLEAVE()

#pragma unroll 1
  for (u32 tcount = 0; tcount < ntile; ++tcount) {
    const u32 e = tab16(p.gt_w, xcd * 64u + slot + tcount * NGX) & 511u;
    const u32 a = e >> 5, b = e & 31u;
    f32x4 acc[4][4];
#pragma unroll
    for (int i = 0; i < 4; ++i)
#pragma unroll
      for (int j = 0; j < 4; ++j) acc[i][j] = f32x4{0.f, 0.f, 0.f, 0.f};
#pragma unroll 1
    for (u32 it = 0; it < 4; ++it) {
      STEP(0) STEP(1) STEP(2)
      STEP(3)
    }
    SCHED();
    const u32 h = 2u * a + wm;
    const u32 yo = OFF_Y + par * (16u << 20) + (h * 32u + b) * 16384u + wn * 2048u;
#pragma unroll
    for (int mt = 0; mt < 4; ++mt)
#pragma unroll
      for (int u = 0; u < 2; ++u) {
        const f32x4 c0 = acc[mt][2 * u], c1 = acc[mt][2 * u + 1];
        const u32x2 s0 = __builtin_amdgcn_permlane16_swap(pkbf(c0[0], c0[1]), pkbf(c1[0], c1[1]), false, false);
        const u32x2 s1 = __builtin_amdgcn_permlane16_swap(pkbf(c0[2], c0[3]), pkbf(c1[2], c1[3]), false, false);
        const u32x4 r = {s0.x, s1.x, s0.y, s1.y};
        __builtin_amdgcn_raw_buffer_store_b128(r, rme, lane * 16u + yo + mt * 4096u + u * 1024u, 0, AUX_SYS);
      }
    asm volatile("s_waitcnt vmcnt(0)" ::: "memory");
    BAR();
    if (tid < 8u && *(volatile const u32 AS3*)(lds + LDS_FAIL) == 0u) {
      const u32 hh = tid >> 2, o = tid & 3u;
      const u32 t0 = (2u * a) >= (u32)NBI, t1 = (2u * a + 1u) >= (u32)NBI;
      const u32 ty = hh ? t1 : t0;
      if (hh == 0 || t1 != t0) {
        const u32 d = ((grp ^ ty) * 4u + o) & 7u;
        char* base = p.ws[0];
#pragma unroll
        for (u32 r = 1; r < 8; ++r) base = d == r ? p.ws[r] : base;
        __hip_atomic_store((u32*)(base + OFF_YF + (ty * 512u + e) * 16u + q * 4u), seq, __ATOMIC_RELAXED,
                           __HIP_MEMORY_SCOPE_SYSTEM);
      }
    }
    STAMP(4 + 4 * tcount);
  }
}

void build_tables(Params& p) {
  int pos[48], n = 0;
  const int AI = (NBI + 1) / 2;
  for (int i = 0; i < AI; ++i) {
    p.send[n] = (unsigned char)(16 + i); pos[16 + i] = n++;
    p.send[n] = (unsigned char)i; pos[i] = n++;
  }
  for (int a = AI; a < 16; ++a) { p.send[n] = (unsigned char)a; pos[a] = n++; }
  for (int b = AI; b < 32; ++b) { p.send[n] = (unsigned char)(16 + b); pos[16 + b] = n++; }
  int nx[8] = {0, 0, 0, 0, 0, 0, 0, 0}, ni = 0, nc = NUI;
  for (int s = 0; s < 48; ++s)
    for (int a = 0; a < 16; ++a)
      for (int b = 0; b < 32; ++b) {
        const int av = pos[a] > pos[16 + b] ? pos[a] : pos[16 + b];
        if (av != s) continue;
        const int k = (a & 1) + 2 * (b & 3);
        p.gt[k][nx[k]++] = (unsigned short)(a * 32 + b + (s << 9));
        for (int hh = 0; hh < 2; ++hh) {
          const int h = 2 * a + hh;
          if (h < NBI) p.ru[ni++] = (unsigned short)(h * 32 + b);
          else p.ru[nc++] = (unsigned short)(h * 32 + b);
        }
      }
}

}

extern "C" void kernel_launch(int rank, int nranks, ncclComm_t comm, hipStream_t stream,
                              void* const* d_in, const int* in_sizes, int n_in,
                              void* d_out, int out_size, const int* in_dtypes, int out_dtype,
                              void* const* d_out_all, void* const* d_ws_all, size_t ws_bytes) {
  (void)comm; (void)in_sizes; (void)n_in; (void)out_size; (void)in_dtypes; (void)out_dtype; (void)d_out_all;
  if (nranks != NRANK || (unsigned)rank >= (unsigned)NRANK || ws_bytes < WS_USED) {
    fprintf(stderr, "unsupported configuration\n");
    return;
  }
  Params p;
  p.x = (const char*)d_in[0];
  p.w = (const char*)d_in[1];
  p.out = (char*)d_out;
  for (int r = 0; r < NRANK; ++r) p.ws[r] = (char*)d_ws_all[r];
  p.rank = rank;
  p.pad = 0;
  build_tables(p);
  fused_gemm_rs<<<dim3(NWG), dim3(512), 0, stream>>>(p);
  HIP_CHECK(hipGetLastError());
}
```

```cpp
#include <hip/hip_runtime.h>
#include <rccl/rccl.h>
#include <cstdio>
#include <cstdint>

#define HIP_CHECK(expr) do { hipError_t _e = (expr); if (_e != hipSuccess) { \
    fprintf(stderr, "HIP error %s at %s:%d\n", hipGetErrorString(_e), __FILE__, __LINE__); \
    return; } } while (0)

namespace {

typedef unsigned int u32;
typedef unsigned long long u64;
typedef u32 u32x4 __attribute__((ext_vector_type(4)));
typedef u32 u32x2 __attribute__((ext_vector_type(2)));
typedef float f32x4 __attribute__((ext_vector_type(4)));
typedef float f32x2 __attribute__((ext_vector_type(2)));
typedef short s16x4 __attribute__((ext_vector_type(4)));
typedef short s16x8 __attribute__((ext_vector_type(8)));
typedef __bf16 bf16x8 __attribute__((ext_vector_type(8)));
typedef __bf16 bf16x2 __attribute__((ext_vector_type(2)));
#define AS3 __attribute__((address_space(3)))

#define CFG_NBC 12
#define CFG_NPUSH 16
#define CFG_NRED 16

#define CFG_GATE 40
#define CFG_PW 4
#define CFG_CW 4
#define CFG_NPS (CFG_NPUSH * CFG_PW / 4)
#define CFG_RG 4
#define CFG_RW 4
#define CFG_QS 15
#define CFG_PS -1
#define CFG_XPOS 4
#define CFG_QOWN 3.8
#define CFG_QREM 1.5
#define CFG_ROWN 4.2
#define CFG_RREM 4.8
#define CFG_A0 3.4
#define CFG_A1 0.6
#define CFG_OUT_AUX 0
#define CFG_STG0 50u
#define CFG_STG 7u
constexpr int NRANK = 8;
constexpr int PW = CFG_PW, RW = CFG_RW, CW = CFG_CW;
constexpr int NBC = CFG_NBC;
constexpr int NBI = 32 - NBC;
constexpr int RC = 16 * NBC;
constexpr int NWG = 256;
constexpr int NPUSH = CFG_NPUSH;
constexpr int NRED = CFG_NRED;
constexpr int NGX = (NWG - NPUSH - NRED) / 8;
static_assert((NPUSH + NRED) % 8 == 0, "roles must keep the XCD round-robin aligned");
constexpr int NGEMM = NGX * 8;
constexpr int NTI = (NBI + 1) / 2 * 32, NTC = (16 - NBI / 2) * 32;
constexpr int ITMAX = 832;

constexpr u32 OFF_CTR = 0;
constexpr u32 OFF_TMO = 8192;
constexpr u32 OFF_IF = 16384;
constexpr u32 OFF_YF = 32768;
constexpr u32 OFF_DBG = 65536;
constexpr u32 OFF_Y = 1u << 20;
constexpr u32 OFF_XR = 33u << 20;
constexpr u32 OFF_WR = 37u << 20;
constexpr u32 WS_USED = 45u << 20;
constexpr u64 DEADLINE = 50000000ull;
constexpr int AUX_SYS = 17;
constexpr u32 SRD3 = 0x00020000;

struct Params {
  const char* x;
  const char* w;
  char* out;
  char* ws[NRANK];
  int rank;
  int pad;
  union { unsigned char send[48]; u32 send_w[12]; };
  u32 wg[NGEMM];
  union { unsigned short it[ITMAX]; u32 it_w[ITMAX / 2]; };
  union { unsigned short rt[NTI + NTC]; u32 rt_w[(NTI + NTC) / 2]; };
};
__device__ __forceinline__ u32 tab8(const u32* t, u32 i) { return (t[i >> 2] >> ((i & 3u) * 8u)) & 0xffu; }
__device__ __forceinline__ u32 tab16(const u32* t, u32 i) { return (t[i >> 1] >> ((i & 1u) * 16u)) & 0xffffu; }

#define WAIT_VM0() do { asm volatile("" ::: "memory"); __builtin_amdgcn_s_waitcnt(0x0F70); asm volatile("" ::: "memory"); } while (0)
#define WAIT_LGKM0() do { asm volatile("" ::: "memory"); __builtin_amdgcn_s_waitcnt(0xC07F); asm volatile("" ::: "memory"); } while (0)
__device__ __forceinline__ u32 rfl(u32 v) { return __builtin_amdgcn_readfirstlane(v); }
__device__ __forceinline__ __amdgpu_buffer_rsrc_t mkrs(const void* p, u32 bytes) {
  return __builtin_amdgcn_make_buffer_rsrc((void*)p, (short)0, (int)bytes, SRD3);
}
__device__ __forceinline__ u32 vrow0(u32 grp, u32 vb) {
  const u32 h = vb >> 2, o = vb & 3;
  return h < (u32)NBI ? 512u * (4u * grp + o) + RC + 16u * h : 512u * (4u * (grp ^ 1u) + o) + 16u * (h - NBI);
}

#define STAMP(idx) do {} while (0)
#define STAMPW(idx) do {} while (0)

__device__ __forceinline__ bool wait_words(const u32* word, u32* tmo, u32 seq) {
  const u64 t0 = __builtin_amdgcn_s_memrealtime();
  for (u32 it = 0;; ++it) {
    const u32 v = __hip_atomic_load(word, __ATOMIC_RELAXED, __HIP_MEMORY_SCOPE_SYSTEM);
    if (__builtin_amdgcn_ballot_w64(v < seq) == 0) return true;
    if ((it & 7u) == 7u) {
      const u32 t = __hip_atomic_load(tmo, __ATOMIC_RELAXED, __HIP_MEMORY_SCOPE_AGENT);
      if (rfl(t) == seq || __builtin_amdgcn_s_memrealtime() - t0 > DEADLINE) {
        __hip_atomic_store(tmo, seq, __ATOMIC_RELAXED, __HIP_MEMORY_SCOPE_AGENT);
        return false;
      }
    }
    __builtin_amdgcn_s_sleep(2);
  }
}

#define CFG_GLDS_BITS "sc0 sc1"
template <u32 IMM>
__device__ __forceinline__ void glds(u32 m0v, u32 voff, __amdgpu_buffer_rsrc_t rs, u32 soff) {
  u32 keep;
  asm volatile("s_mov_b32 %0, m0\n\ts_add_u32 m0, %1, %5\n\ts_nop 2\n\t"
               "buffer_load_dwordx4 %2, %3, %4 offen " CFG_GLDS_BITS " lds\n\ts_mov_b32 m0, %0"
               : "=&s"(keep) : "s"(m0v), "v"(voff), "s"(rs), "s"(soff), "i"(IMM) : "memory", "scc");
}

__device__ __forceinline__ float bflo(u32 v) { return __builtin_bit_cast(float, v << 16); }
__device__ __forceinline__ float bfhi(u32 v) { return __builtin_bit_cast(float, v & 0xffff0000u); }
__device__ __forceinline__ u32 pkbf(float a, float b) {
  f32x2 f = {a, b};
  return __builtin_bit_cast(u32, __builtin_convertvector(f, bf16x2));
}

__device__ __forceinline__ void reduce_units(const Params& p, u32 rank, char* me, u32 seq, u32 type, u32 tbase,
                                             u32 count, u32 first, u32 step, u32 lane) {
  const u32 grp = rank >> 2, q = rank & 3;
  u32* tmo = (u32*)(me + OFF_TMO);
  const u32 pg = (grp ^ type) * 4u;
  const char* pp0 = p.ws[(pg + ((q + 1) & 3)) & 7];
  const char* pp1 = p.ws[(pg + ((q + 2) & 3)) & 7];
  const char* pp2 = p.ws[(pg + ((q + 3) & 3)) & 7];
  const char* pp3 = p.ws[(pg + q) & 7];
  __amdgpu_buffer_rsrc_t rme = mkrs(me, WS_USED);
  __amdgpu_buffer_rsrc_t ro = mkrs(p.out, 512u * 8192u);
  const u32 ybase = OFF_Y + (seq & 1u) * (16u << 20);
  const u32 g = lane >> 4;
  const u32 vo = (lane & 15u) * 8192u + (16u * (g & 1u) + 8u * (g >> 1)) * 2u;
  const u32 vl = lane * 16u;
  const u32 vf = ((lane >> 1) & 3u) * 16u + (lane & 1u) * 4u;
  if (first >= count) return;
  u32 e = tab16(p.rt_w, tbase + (first >> 1));
  u32 fl = __builtin_amdgcn_raw_buffer_load_b32(rme, vf, OFF_YF + (e & 511u) * 64u + (first & 1u) * 8u, AUX_SYS);
  for (u32 u = first; u < count; u += step) {
    const u32 h = ((e >> 5) & 15u) * 2u + (u & 1u), b = e & 31u;
    const bool mine = (h >= (u32)NBI) == (type != 0u);
    bool ok = true;
    if (mine && __builtin_amdgcn_ballot_w64(fl < seq) != 0)
      ok = wait_words((const u32*)(me + OFF_YF + (e & 511u) * 64u + (u & 1u) * 8u + vf), tmo, seq);
    const u32 un = u + step < count ? u + step : u;
    const u32 en = tab16(p.rt_w, tbase + (un >> 1));
    fl = __builtin_amdgcn_raw_buffer_load_b32(rme, vf, OFF_YF + (en & 511u) * 64u + (un & 1u) * 8u, AUX_SYS);
    const bool pushed = (e >> 11) & 1u;
    const u32 so = ybase + (h * 32u + b) * 16384u;
    const u32 so0 = so + (pushed ? 0u : q * 4096u), so1 = so + (pushed ? 4096u : q * 4096u);
    const u32 so2 = so + (pushed ? 8192u : q * 4096u), so3 = so + (pushed ? 12288u : q * 4096u);
    const __amdgpu_buffer_rsrc_t rp0 = mkrs(pushed ? me : pp0, WS_USED), rp1 = mkrs(pushed ? me : pp1, WS_USED);
    const __amdgpu_buffer_rsrc_t rp2 = mkrs(pushed ? me : pp2, WS_USED), rp3 = mkrs(pushed ? me : pp3, WS_USED);
    const u32 orow = type == 0 ? RC + 16u * h : 16u * (h - NBI);
    const u32 oo = orow * 8192u + b * 256u;
    if (!mine) {
    } else if (ok) {
      u32x4 d[4][4];
#pragma unroll
      for (int i = 0; i < 4; ++i) {
        d[0][i] = __builtin_amdgcn_raw_buffer_load_b128(rp0, vl, so0 + i * 1024u, AUX_SYS);
        d[1][i] = __builtin_amdgcn_raw_buffer_load_b128(rp1, vl, so1 + i * 1024u, AUX_SYS);
        d[2][i] = __builtin_amdgcn_raw_buffer_load_b128(rp2, vl, so2 + i * 1024u, AUX_SYS);
        d[3][i] = __builtin_amdgcn_raw_buffer_load_b128(rp3, vl, so3 + i * 1024u, AUX_SYS);
      }
#pragma unroll
      for (int i = 0; i < 4; ++i) {
        u32x4 r;
#pragma unroll
        for (int c = 0; c < 4; ++c) {
          const float lo = (bflo(d[0][i][c]) + bflo(d[1][i][c])) + (bflo(d[2][i][c]) + bflo(d[3][i][c]));
          const float hi = (bfhi(d[0][i][c]) + bfhi(d[1][i][c])) + (bfhi(d[2][i][c]) + bfhi(d[3][i][c]));
          r[c] = pkbf(fmaxf(lo, 0.f), fmaxf(hi, 0.f));
        }
        __builtin_amdgcn_raw_buffer_store_b128(r, ro, vo + oo + i * 64u, 0, CFG_OUT_AUX);
      }
    } else {
      const u32x4 nanv = {0x7fc07fc0u, 0x7fc07fc0u, 0x7fc07fc0u, 0x7fc07fc0u};
#pragma unroll
      for (int i = 0; i < 4; ++i) __builtin_amdgcn_raw_buffer_store_b128(nanv, ro, vo + oo + i * 64u, 0, 0);
    }
    e = en;
  }
}

struct Frag { bf16x8 x[4]; bf16x8 w[4]; };

template <int STAGE, int KS>
__device__ __forceinline__ void read_frag(Frag& f, const char AS3* lds, u32 xa0, u32 xa1, const u32 (&wa)[4]) {
  const u32 xa = KS ? xa1 : xa0;
  f.x[0] = *(const bf16x8 AS3*)(lds + xa + STAGE * 32768);
#pragma unroll
  for (int nt = 0; nt < 4; ++nt) {
    const s16x4 lo = __builtin_amdgcn_ds_read_tr16_b64_v4i16((s16x4 AS3*)(lds + wa[nt] + STAGE * 32768 + KS * 8192));
    const s16x4 hi = __builtin_amdgcn_ds_read_tr16_b64_v4i16((s16x4 AS3*)(lds + wa[nt] + STAGE * 32768 + KS * 8192 + 1024));
    const s16x8 v = __builtin_shufflevector(lo, hi, 0, 1, 2, 3, 4, 5, 6, 7);
    f.w[nt] = __builtin_bit_cast(bf16x8, v);
  }
#pragma unroll
  for (int mt = 1; mt < 4; ++mt)
    f.x[mt] = *(const bf16x8 AS3*)(lds + xa + STAGE * 32768 + mt * 2048);
}

#define SCHED() __builtin_amdgcn_sched_barrier(0)
#define BAR() __builtin_amdgcn_s_barrier()
constexpr u32 LDS_SEQ = 131072, LDS_FAIL = 131076;

__global__ void __launch_bounds__(512) fused_gemm_rs(const Params p) {
  __shared__ __attribute__((aligned(1024))) char lds_raw[131072 + 64];
  const u32 tid = threadIdx.x, lane = tid & 63u;
  const u32 wave = rfl(tid >> 6);
  const u32 rank = (u32)p.rank & 7u, grp = rank >> 2, q = rank & 3u;
  char* me = p.ws[rank];
  u32* tmo = (u32*)(me + OFF_TMO);
  u32* ctr = (u32*)(me + OFF_CTR) + blockIdx.x * 8u + wave;
  const u32 bid = blockIdx.x;
  STAMP(0);

  if (bid < (u32)(NPUSH + NRED)) {
    const bool cross = bid < (u32)NPUSH;
    if (wave >= (cross ? (u32)(PW > CW ? PW : CW) : (u32)RW)) return;
    u32 seq;
    if (cross) {
      constexpr u32 NPS = CFG_NPS;
      const u32 pw = bid * PW + wave, dq = pw & 3u, ps0 = pw >> 2;
      const u32 ps = (wave < (u32)PW && ps0 < NPS) ? ps0 : 384u;
      const u32 dst = ((grp ^ 1u) * 4u + dq) & 7u;
      __amdgpu_buffer_rsrc_t rd = mkrs(p.ws[dst], WS_USED);
      const u32 lr = lane >> 4, lc = (lane & 15u) * 16u;
      const u32 vx = lr * 1024u + lc, vw = lr * 8192u + lc;
      u32x4 v[4];
      u32 dso = 0, dvo = 0, dstep = 0, dfl = 0;
      auto prep = [&](u32 gi, u32x4 (&r)[4], u32& o_so, u32& o_vo, u32& o_step, u32& o_fl) {
        const u32 c = tab8(p.send_w, gi >> 3), part = gi & 7u;
        const bool isx = c < 16u;
        const u32 vb = c * 8u + part, b = c - 16u;
        const u32 sso = isx ? vrow0(grp ^ 1u, vb) * 1024u + dq * 256u : (128u * dq + 16u * part) * 8192u + b * 256u;
        o_so = isx ? OFF_XR + vb * 16384u + q * 256u : OFF_WR + (128u * q + 16u * part) * 8192u + b * 256u;
        o_vo = isx ? vx : vw;
        o_step = isx ? 4096u : 32768u;
        o_fl = OFF_IF + c * 256u + (q * 8u + part) * 4u;
        __amdgpu_buffer_rsrc_t rs = mkrs(isx ? p.x : p.w, 4u << 20);
#pragma unroll
        for (int i = 0; i < 4; ++i) r[i] = __builtin_amdgcn_raw_buffer_load_b128(rs, o_vo, sso + i * o_step, 0);
      };
      const u64 tstart = __builtin_amdgcn_s_memrealtime();
      prep(ps < 384u ? ps : 0u, v, dso, dvo, dstep, dfl);
      asm volatile("" ::: "memory");
      seq = rfl(__hip_atomic_load(ctr, __ATOMIC_RELAXED, __HIP_MEMORY_SCOPE_SYSTEM) + 1u);
      WAIT_VM0();
      while (__builtin_amdgcn_s_memrealtime() - tstart < (u64)(CFG_STG0 + ps0 * CFG_STG)) __builtin_amdgcn_s_sleep(1);
      const u32 par = seq & 1u;
      const u32 fvo = lane == 0 ? 0u : 0x80000000u;
      for (u32 gi = ps; gi < 384u; gi += NPS) {
        u32x4 vn[4];
        u32 nso, nvo, nstep, nfl;
        const u32 pso = dso + par * (dstep == 4096u ? (2u << 20) : (4u << 20));
#pragma unroll
        for (int i = 0; i < 4; ++i) __builtin_amdgcn_raw_buffer_store_b128(v[i], rd, dvo + pso + i * dstep, 0, AUX_SYS);
        asm volatile("" ::: "memory");
        const u32 gn = gi + NPS < 384u ? gi + NPS : gi;
        prep(gn, vn, nso, nvo, nstep, nfl);
        WAIT_VM0();
        __builtin_amdgcn_raw_buffer_store_b32(seq, rd, fvo, dfl, AUX_SYS);
#pragma unroll
        for (int i = 0; i < 4; ++i) v[i] = vn[i];
        dso = nso; dvo = nvo; dstep = nstep; dfl = nfl;
      }
      STAMPW(1 + (wave & 3));
      const u32 c = tab8(p.send_w, CFG_GATE + (lane >> 5));
      (void)wait_words((const u32*)(me + OFF_IF + c * 256u) + (lane & 31u), tmo, seq);
    } else {
      seq = rfl(__hip_atomic_load(ctr, __ATOMIC_RELAXED, __HIP_MEMORY_SCOPE_SYSTEM) + 1u);
    }
    __hip_atomic_store(ctr, seq, __ATOMIC_RELAXED, __HIP_MEMORY_SCOPE_SYSTEM);
    reduce_units(p, rank, me, seq, cross ? 1u : 0u, cross ? NTI : 0u, cross ? 2 * NTC : 2 * NTI,
                 cross ? (wave < (u32)CW ? bid * CW + wave : 4096u) : (bid - NPUSH) * RW + wave, cross ? NPUSH * CW : NRED * RW, lane);
    STAMPW(5 + (wave & 3));
    return;
  }

  const char AS3* lds = (const char AS3*)lds_raw;
  const u32 gj = bid - (NPUSH + NRED), slot = gj >> 3;
  const u32 wgw = p.wg[gj];
  const u32 ioff = wgw & 1023u, nq = (wgw >> 10) & 7u, ntile = (wgw >> 13) & 7u;
  const u32 g = lane >> 4, l15 = lane & 15u;
  __amdgpu_buffer_rsrc_t rme = mkrs(me, WS_USED);

  if (wave >= 4u) {
    const u64 tstart = __builtin_amdgcn_s_memrealtime();
    __builtin_amdgcn_s_setprio(2);
    const u32 lw = wave - 4u;
    u32 vX[2], vW[2];
#pragma unroll
    for (u32 j = 0; j < 2; ++j) {
      vX[j] = (8u * j + (lane >> 3)) * 1024u + (((lane & 7u) ^ (4u * j + g)) * 16u);
      vW[j] = g * 8192u + ((((l15 >> 1) ^ (g | (j << 2))) * 2u + (l15 & 1u)) * 16u);
    }
    const u32 m0b = (u32)(uintptr_t)lds + lw * 4096u;
    const u32 rot = (slot * 3u) & 7u;
    const __amdgpu_buffer_rsrc_t rxo = mkrs(p.x, 4u << 20), rwo = mkrs(p.w, 4u << 20);
    const u32 total = ntile * 16u;
    u32 issued = 0, seq = 0;
    u32 sx0 = 0, sx1 = 0, sw = 0, ea = 0, eb = 0;
    bool rem = false;
    {
      const u32 dl = (wgw >> 16) * 4u;
      while (__builtin_amdgcn_s_memrealtime() - tstart < (u64)dl) __builtin_amdgcn_s_sleep(4);
    }
    if (nq) {
      u32 vWq[2];
#pragma unroll
      for (u32 j = 0; j < 2; ++j) {
        const u32 gr = (l15 >> 1) ^ (g | (j << 2));
        vWq[j] = g * 8192u + (gr >> 2) * (64u * 8192u) + (gr & 3u) * 32u + (l15 & 1u) * 16u;
      }
      auto ldx = [&](u32 kb, const __amdgpu_buffer_rsrc_t& r, u32 rt) {
        const u32 m0s = m0b + kb * 32768u, kx = ((kb + rt) & 3u) * 256u;
        glds<0>(m0s, vX[0], r, sx0 + kx);
        glds<1024>(m0s, vX[1], r, sx0 + kx);
        glds<2048>(m0s, vX[0], r, sx1 + kx);
        glds<3072>(m0s, vX[1], r, sx1 + kx);
      };
      auto ldw = [&](u32 kb, const __amdgpu_buffer_rsrc_t& r, u32 rt) {
        const u32 m0s = m0b + kb * 32768u, kw = sw + ((kb + rt) & 3u) * (128u * 8192u);
        glds<16384>(m0s, vWq[0], r, kw);
        glds<16384 + 1024>(m0s, vWq[0], r, kw + 32768u);
        glds<16384 + 2048>(m0s, vWq[1], r, kw + 65536u);
        glds<16384 + 3072>(m0s, vWq[1], r, kw + 98304u);
      };
#pragma unroll 1
      for (u32 i = 0; i < nq; ++i) {
        const u32 e = tab16(p.it_w, ioff + i);
        ea = (e >> 5) & 15u; eb = e & 31u;
        const u32 sub = (e >> 9) & 3u, hm = sub >> 1, hn = sub & 1u;
        const u32 vb = ea * 8u + hm * 4u + (lw & 1u) * 2u;
        sx0 = vrow0(grp, vb) * 1024u + (lw >> 1) * 128u; sx1 = vrow0(grp, vb + 1u) * 1024u + (lw >> 1) * 128u;
        sw = lw * 16u * 8192u + eb * 256u + hn * 128u;
#pragma unroll 1
        for (u32 kb = 0; kb < 4; ++kb) { ldx(kb, rxo, sub); ldw(kb, rwo, sub); }
        asm volatile("s_waitcnt vmcnt(24)" ::: "memory"); BAR();
        asm volatile("s_waitcnt vmcnt(16)" ::: "memory"); BAR();
        asm volatile("s_waitcnt vmcnt(8)" ::: "memory"); BAR();
        asm volatile("s_waitcnt vmcnt(0)" ::: "memory"); BAR();
        BAR();
        seq = rfl(*(volatile const u32 AS3*)(lds + LDS_SEQ));
        const u32 par = seq & 1u;
        sx0 = OFF_XR + par * (2u << 20) + vb * 16384u + (lw >> 1) * 128u; sx1 = sx0 + 16384u;
        sw += OFF_WR + par * (4u << 20);
        const u32* fx = (const u32*)(me + OFF_IF + ea * 256u) + ((lane >> 2) & 3u) * 8u + hm * 4u + (lane & 3u);
        const u32* fw = (const u32*)(me + OFF_IF + (16u + eb) * 256u) + (lane & 31u);
        const u32 f0 = __hip_atomic_load(lane < 32u ? fw : fx, __ATOMIC_RELAXED, __HIP_MEMORY_SCOPE_SYSTEM);
        if (__builtin_amdgcn_ballot_w64(f0 < seq) == 0) {
#pragma unroll 1
          for (u32 kb = 0; kb < 4; ++kb) { ldx(kb, rme, sub); ldw(kb, rme, sub); }
          asm volatile("s_waitcnt vmcnt(24)" ::: "memory"); BAR();
          asm volatile("s_waitcnt vmcnt(16)" ::: "memory"); BAR();
          asm volatile("s_waitcnt vmcnt(8)" ::: "memory"); BAR();
        } else {
          bool ok = true;
#pragma unroll 1
          for (u32 ph = 0; ph < 2; ++ph) {
            const bool isx = ((e >> 12) & 1u) != ph;
            ok = wait_words(isx ? fx : fw, tmo, seq) && ok;
            if (isx) {
#pragma unroll 1
              for (u32 kb = 0; kb < 4; ++kb) ldx(kb, rme, sub);
            } else {
#pragma unroll 1
              for (u32 kb = 0; kb < 4; ++kb) ldw(kb, rme, sub);
            }
          }
          if (!ok) *(volatile u32 AS3*)(lds + LDS_FAIL) = seq;
          asm volatile("s_waitcnt vmcnt(12) lgkmcnt(0)" ::: "memory"); BAR();
          asm volatile("s_waitcnt vmcnt(8)" ::: "memory"); BAR();
          asm volatile("s_waitcnt vmcnt(4)" ::: "memory"); BAR();
        }
        asm volatile("s_waitcnt vmcnt(0)" ::: "memory"); BAR();
        BAR();
      }
    }
    for (int n = -1; n < (int)total; ++n) {
      while ((int)issued <= n + 3 && issued < total) {
        const u32 st = issued & 15u;
        if (st == 0u) {
          const u32 e = tab16(p.it_w, ioff + nq + (issued >> 4));
          ea = (e >> 5) & 15u; eb = e & 31u;
          const u32 vb = ea * 8u + lw * 2u;
          sx0 = vrow0(grp, vb) * 1024u; sx1 = vrow0(grp, vb + 1u) * 1024u;
          sw = lw * 16u * 8192u + eb * 256u;
          rem = false;
        } else if (st == 8u) {
          seq = rfl(*(volatile const u32 AS3*)(lds + LDS_SEQ));
          const u32* fw = (const u32*)(me + OFF_IF + (lane < 32u ? ea : 16u + eb) * 256u) + (lane & 31u);
          if ((int)issued <= n + 1) {
            if (!wait_words(fw, tmo, seq)) *(volatile u32 AS3*)(lds + LDS_FAIL) = seq;
          } else {
            const u32 v = __hip_atomic_load(fw, __ATOMIC_RELAXED, __HIP_MEMORY_SCOPE_SYSTEM);
            if (__builtin_amdgcn_ballot_w64(v < seq) != 0) break;
          }
          const u32 par = seq & 1u;
          sx0 = OFF_XR + par * (2u << 20) + (ea * 8u + lw * 2u) * 16384u; sx1 = sx0 + 16384u;
          sw = OFF_WR + par * (4u << 20) + lw * 16u * 8192u + eb * 256u;
          rem = true;
        }
        const __amdgpu_buffer_rsrc_t rx = rem ? rme : rxo, rw = rem ? rme : rwo;
        const u32 kk = (issued + rot) & 7u;
        const u32 m0s = m0b + (issued & 3u) * 32768u;
        const u32 kx = kk * 128u, kw = sw + kk * (64u * 8192u);
        glds<0>(m0s, vX[0], rx, sx0 + kx);
        glds<1024>(m0s, vX[1], rx, sx0 + kx);
        glds<16384>(m0s, vW[0], rw, kw);
        glds<16384 + 1024>(m0s, vW[0], rw, kw + 32768u);
        glds<2048>(m0s, vX[0], rx, sx1 + kx);
        glds<3072>(m0s, vX[1], rx, sx1 + kx);
        glds<16384 + 2048>(m0s, vW[1], rw, kw + 65536u);
        glds<16384 + 3072>(m0s, vW[1], rw, kw + 98304u);
        ++issued;
        if (n < 0 && issued == 2u) break;
      }
      const int ahead = (int)issued - (n + 2);
      if (ahead >= 2) asm volatile("s_waitcnt vmcnt(16) lgkmcnt(0)" ::: "memory");
      else if (ahead == 1) asm volatile("s_waitcnt vmcnt(8) lgkmcnt(0)" ::: "memory");
      else asm volatile("s_waitcnt vmcnt(0) lgkmcnt(0)" ::: "memory");
      BAR();
      if ((n & 15) == 15 && n >= 0) BAR();
    }
    return;
  }

  const u32 seq = rfl(__hip_atomic_load(ctr, __ATOMIC_RELAXED, __HIP_MEMORY_SCOPE_SYSTEM) + 1u);
  __hip_atomic_store(ctr, seq, __ATOMIC_RELAXED, __HIP_MEMORY_SCOPE_SYSTEM);
  if (wave == 0) {
    *(volatile u32 AS3*)(lds + LDS_SEQ) = seq;
    *(volatile u32 AS3*)(lds + LDS_FAIL) = 0u;
  }
  const u32 par = seq & 1u;
  const u32 wm = wave >> 1, wn = wave & 1u;
  const u32 xa0 = (64u * wm + l15) * 128u + ((g ^ (l15 >> 1)) * 16u);
  const u32 xa1 = xa0 ^ 64u;
  u32 wa[4];
  {
    const u32 qq = l15 >> 2, pp = lane & 3u, f = qq | ((g & 1u) << 2);
#pragma unroll
    for (u32 nt = 0; nt < 4; ++nt) wa[nt] = 16384u + (8u * g + qq) * 256u + (((4u * wn + nt) ^ f) * 32u) + 8u * pp;
  }
  WAIT_LGKM0();
  if (nq) {
    const u32 xq0 = (16u * wave + l15) * 128u + ((g ^ (l15 >> 1)) * 16u), xq1 = xq0 ^ 64u;
    u32 wq[2][4];
#pragma unroll
    for (u32 nt = 0; nt < 4; ++nt) { wq[0][nt] = wa[nt] ^ (wn * 128u); wq[1][nt] = wq[0][nt] ^ 128u; }
#pragma unroll 1
    for (u32 i = 0; i < nq; ++i) {
      const u32 e = tab16(p.it_w, ioff + i);
      f32x4 acc[4];
#pragma unroll
      for (int j = 0; j < 4; ++j) acc[j] = f32x4{0.f, 0.f, 0.f, 0.f};
#pragma unroll 1
      for (u32 half = 0; half < 2; ++half) {
#pragma unroll
        for (int kb = 0; kb < 4; ++kb) {
          SCHED();
          BAR();
          SCHED();
          bf16x8 xf[4], wf[4][4];
#pragma unroll
          for (int ks = 0; ks < 4; ++ks) {
            xf[ks] = *(const bf16x8 AS3*)(lds + ((ks & 1) ? xq1 : xq0) + (ks >> 1) * 8192 + kb * 32768);
#pragma unroll
            for (int nt = 0; nt < 4; ++nt) {
              const char AS3* ad = lds + wq[ks >> 1][nt] + kb * 32768 + (ks & 1) * 8192;
              const s16x4 lo = __builtin_amdgcn_ds_read_tr16_b64_v4i16((s16x4 AS3*)ad);
              const s16x4 hi = __builtin_amdgcn_ds_read_tr16_b64_v4i16((s16x4 AS3*)(ad + 1024));
              wf[ks][nt] = __builtin_bit_cast(bf16x8, __builtin_shufflevector(lo, hi, 0, 1, 2, 3, 4, 5, 6, 7));
            }
          }
          SCHED();
#pragma unroll
          for (int ks = 0; ks < 4; ++ks)
#pragma unroll
            for (int nt = 0; nt < 4; ++nt)
              acc[nt] = __builtin_amdgcn_mfma_f32_16x16x32_bf16(wf[ks][nt], xf[ks], acc[nt], 0, 0, 0);
        }
        SCHED();
        WAIT_LGKM0();
        BAR();
        STAMP(1 + 2 * half + 4 * i);
      }
      const u32 sub = (e >> 9) & 3u, h = ((e >> 5) & 15u) * 2u + (sub >> 1);
      const bool push = (e >> 11) & 1u;
      const u32 own = ((grp ^ (h >= (u32)NBI ? 1u : 0u)) * 4u + wave) & 7u;
      const __amdgpu_buffer_rsrc_t rown = mkrs(p.ws[own], WS_USED);
      const __amdgpu_buffer_rsrc_t rdst = mkrs(push ? p.ws[own] : me, WS_USED);
      const u32 yo = OFF_Y + par * (16u << 20) + (h * 32u + (e & 31u)) * 16384u + (push ? q : wave) * 4096u + (sub & 1u) * 2048u;
#pragma unroll
      for (int u = 0; u < 2; ++u) {
        const f32x4 c0 = acc[2 * u], c1 = acc[2 * u + 1];
        const u32x2 s0 = __builtin_amdgcn_permlane16_swap(pkbf(c0[0], c0[1]), pkbf(c1[0], c1[1]), false, false);
        const u32x2 s1 = __builtin_amdgcn_permlane16_swap(pkbf(c0[2], c0[3]), pkbf(c1[2], c1[3]), false, false);
        const u32x4 r = {s0.x, s1.x, s0.y, s1.y};
        __builtin_amdgcn_raw_buffer_store_b128(r, rdst, lane * 16u + yo + u * 1024u, 0, AUX_SYS);
      }
      const u32 fail = *(volatile const u32 AS3*)(lds + LDS_FAIL);
      WAIT_VM0();
      __builtin_amdgcn_raw_buffer_store_b32(seq, rown, (lane == 0 && fail == 0u) ? 0u : 0x80000000u,
                                            OFF_YF + (e & 511u) * 64u + q * 16u + sub * 4u, AUX_SYS);
      STAMP(4 + 4 * i);
    }
  }
  BAR();
  Frag f0, f1;
  read_frag<0, 0>(f0, lds, xa0, xa1, wa);

#define MMA16(F) \
  _Pragma("unroll") for (int mt = 0; mt < 4; ++mt) \
    _Pragma("unroll") for (int nt = 0; nt < 4; ++nt) \
      acc[mt][nt] = __builtin_amdgcn_mfma_f32_16x16x32_bf16(F.w[nt], F.x[mt], acc[mt][nt], 0, 0, 0);
#define INTERLEAVE() \
  _Pragma("unroll") for (int i = 0; i < 6; ++i) { \
    __builtin_amdgcn_sched_group_barrier(0x008, 1, 0); \
    __builtin_amdgcn_sched_group_barrier(0x100, 2, 0); \
  } \
  __builtin_amdgcn_sched_group_barrier(0x008, 10, 0);
#define STEP(S) \
  SCHED(); \
  read_frag<(S), 1>(f1, lds, xa0, xa1, wa); \
  MMA16(f0) \
  INTERLEAVE() \
  SCHED(); \
  WAIT_LGKM0(); \
  BAR(); \
  SCHED(); \
  read_frag<((S) + 1) & 3, 0>(f0, lds, xa0, xa1, wa); \
  MMA16(f1) \
  INTERLEAVE()

#pragma unroll 1
  for (u32 tcount = 0; tcount < ntile; ++tcount) {
    const u32 e = tab16(p.it_w, ioff + nq + tcount) & 511u;
    const u32 a = e >> 5, b = e & 31u;
    f32x4 acc[4][4];
#pragma unroll
    for (int i = 0; i < 4; ++i)
#pragma unroll
      for (int j = 0; j < 4; ++j) acc[i][j] = f32x4{0.f, 0.f, 0.f, 0.f};
#pragma unroll 1
    for (u32 it = 0; it < 4; ++it) {
      STEP(0) STEP(1) STEP(2)
      STEP(3)
    }
    SCHED();
    const u32 h = 2u * a + wm;
    const u32 yo = OFF_Y + par * (16u << 20) + (h * 32u + b) * 16384u + wn * 2048u;
#pragma unroll
    for (int mt = 0; mt < 4; ++mt)
#pragma unroll
      for (int u = 0; u < 2; ++u) {
        const f32x4 c0 = acc[mt][2 * u], c1 = acc[mt][2 * u + 1];
        const u32x2 s0 = __builtin_amdgcn_permlane16_swap(pkbf(c0[0], c0[1]), pkbf(c1[0], c1[1]), false, false);
        const u32x2 s1 = __builtin_amdgcn_permlane16_swap(pkbf(c0[2], c0[3]), pkbf(c1[2], c1[3]), false, false);
        const u32x4 r = {s0.x, s1.x, s0.y, s1.y};
        __builtin_amdgcn_raw_buffer_store_b128(r, rme, lane * 16u + yo + mt * 4096u + u * 1024u, 0, AUX_SYS);
      }
    asm volatile("s_waitcnt vmcnt(0)" ::: "memory");
    BAR();
    if (tid < 8u && *(volatile const u32 AS3*)(lds + LDS_FAIL) == 0u) {
      const u32 hh = tid & 1u, o = tid >> 1;
      const u32 ty = (2u * a + hh) >= (u32)NBI;
      const u32 d = ((grp ^ ty) * 4u + o) & 7u;
      char* base = p.ws[0];
#pragma unroll
      for (u32 r = 1; r < 8; ++r) base = d == r ? p.ws[r] : base;
      __hip_atomic_store((u64*)(base + OFF_YF + e * 64u + q * 16u + hh * 8u), (u64)seq * 0x100000001ull, __ATOMIC_RELAXED,
                         __HIP_MEMORY_SCOPE_SYSTEM);
    }
    STAMP(4 + 4 * (nq + tcount));
  }
}

void build_tables(Params& p) {
  int pos[48], n = 0;
  const int AI = (NBI + 1) / 2;
  auto add = [&](int c) { p.send[n] = (unsigned char)c; pos[c] = n++; };
  for (int i = 0; i < AI; ++i) { add(16 + i); add(i); }
  for (int b = AI; b < AI + CFG_XPOS; ++b) add(16 + b);
  for (int a = AI; a < 16; ++a) add(a);
  for (int b = AI + CFG_XPOS; b < 32; ++b) add(16 + b);
  unsigned short list[NGEMM][8];
  int cq[NGEMM], cr[NGEMM];
  double fr[NGEMM], st[NGEMM];
  for (int i = 0; i < NGEMM; ++i) { cq[i] = cr[i] = 0; fr[i] = 0.0; st[i] = 0.0; }
  int ni = 0, nc = NTI;
  for (int s = 0; s < 48; ++s)
    for (int a = 0; a < 16; ++a)
      for (int b = 0; b < 32; ++b) {
        const int av = pos[a] > pos[16 + b] ? pos[a] : pos[16 + b];
        if (av != s) continue;
        const int k = (a & 1) + 2 * (b & 3);
        const bool small = s <= CFG_QS, push = small && s <= CFG_PS && 2 * a + 1 < NBI;
        const int e = a * 32 + b + (push ? 2048 : 0);
        const double arr = CFG_A0 + CFG_A1 * s, own = small ? CFG_QOWN : CFG_ROWN, rem = small ? CFG_QREM : CFG_RREM;
        for (int sub = 0; sub < (small ? 4 : 1); ++sub) {
          int best = -1;
          double bt = 0.0;
          for (int j = 0; j < NGX; ++j) {
            const int w = j * 8 + k;
            if (cq[w] + cr[w] >= 7 || (small && cr[w])) continue;
            const double t = fr[w] + own > arr ? fr[w] + own : arr;
            if (best < 0 || t < bt - 1e-9 || (t < bt + 1e-9 && fr[w] > fr[best])) { best = w; bt = t; }
          }
          if (cq[best] + cr[best] == 0) st[best] = arr - own - 1.0 > 0.0 ? arr - own - 1.0 : 0.0;
          list[best][cq[best] + cr[best]] = (unsigned short)(e + (sub << 9) + (pos[a] < pos[16 + b] ? 4096 : 0));
          if (small) ++cq[best]; else ++cr[best];
          fr[best] = bt + rem;
        }
        if (2 * a < NBI) p.rt[ni++] = (unsigned short)e;
        if (2 * a + 1 >= NBI) p.rt[nc++] = (unsigned short)e;
      }
  int off = 0;
  for (int w = 0; w < NGEMM; ++w) {
    p.wg[w] = (u32)off | (u32)cq[w] << 10 | (u32)cr[w] << 13 | (u32)(st[w] * 25.0) << 16;
    for (int i = 0; i < cq[w] + cr[w]; ++i) p.it[off++] = list[w][i];
  }
  for (; off < ITMAX; ++off) p.it[off] = 0;
}
static_assert(512 + 3 * ((CFG_QS + 2) / 2) * ((CFG_QS + 1) / 2) <= ITMAX && CFG_QS < 2 * ((NBI + 1) / 2), "item table");
static_assert(sizeof(Params) <= 4096, "kernel arguments");

}

extern "C" void kernel_launch(int rank, int nranks, ncclComm_t comm, hipStream_t stream,
                              void* const* d_in, const int* in_sizes, int n_in,
                              void* d_out, int out_size, const int* in_dtypes, int out_dtype,
                              void* const* d_out_all, void* const* d_ws_all, size_t ws_bytes) {
  (void)comm; (void)in_sizes; (void)n_in; (void)out_size; (void)in_dtypes; (void)out_dtype; (void)d_out_all;
  if (nranks != NRANK || (unsigned)rank >= (unsigned)NRANK || ws_bytes < WS_USED) {
    fprintf(stderr, "unsupported configuration\n");
    return;
  }
  Params p;
  p.x = (const char*)d_in[0];
  p.w = (const char*)d_in[1];
  p.out = (char*)d_out;
  for (int r = 0; r < NRANK; ++r) p.ws[r] = (char*)d_ws_all[r];
  p.rank = rank;
  p.pad = 0;
  build_tables(p);
  fused_gemm_rs<<<dim3(NWG), dim3(512), 0, stream>>>(p);
  HIP_CHECK(hipGetLastError());
}
```

```cpp
#include <hip/hip_runtime.h>
#include <rccl/rccl.h>
#include <cstdio>
#include <cstdint>
#include <cstddef>

#define HIP_CHECK(expr) do { hipError_t _e = (expr); if (_e != hipSuccess) { \
    fprintf(stderr, "HIP error %s at %s:%d\n", hipGetErrorString(_e), __FILE__, __LINE__); \
    return; } } while (0)

namespace {

typedef unsigned int u32;
typedef unsigned long long u64;
typedef u64 u64x4 __attribute__((ext_vector_type(4)));
typedef u64 u64x2 __attribute__((ext_vector_type(2)));
typedef u32 u32x4 __attribute__((ext_vector_type(4)));
typedef u32 u32x2 __attribute__((ext_vector_type(2)));
typedef float f32x4 __attribute__((ext_vector_type(4)));
typedef float f32x2 __attribute__((ext_vector_type(2)));
typedef short s16x4 __attribute__((ext_vector_type(4)));
typedef short s16x8 __attribute__((ext_vector_type(8)));
typedef __bf16 bf16x8 __attribute__((ext_vector_type(8)));
typedef __bf16 bf16x2 __attribute__((ext_vector_type(2)));
#define AS3 __attribute__((address_space(3)))
#define AS1 __attribute__((address_space(1)))

#define CFG_NBC 12
#define CFG_NH 2
#define CFG_DH 5.688
#define CFG_HREM 1.919
#define CFG_HFR 2.277
#define CFG_TREM 2.381
#define CFG_HS 24
#define CFG_HW 12
#define CFG_HSLACK 0.941
#define CFG_OVER 0x180000u
#define CFG_TAILN 0u
#define CFG_BX 0
#define CFG_YPAD 0x40000u
#define CFG_TA 2050u
#define CFG_TAS 0u
#define CFG_NPUSH 16
#define CFG_NRED 16

#define CFG_PW 4
#define CFG_REM_AUX 17
#define CFG_GATE 34u
#define CFG_CW 7
#define CFG_RW 7
#define CFG_QF 3.300
#define CFG_QOWN 1.643
#define CFG_QREM 3.738
#define CFG_RF 5.453
#define CFG_RQ 3.400
#define CFG_ROWN 2.950
#define CFG_RREM 7.015
#define CFG_A0 3.670
#define CFG_AR 0.434
#define CFG_SLC 0.105
#define CFG_DI 11.178
#define CFG_DC 21.527
#define CFG_SLACK 2.880
#define CFG_OUT_AUX 17
#define CFG_LPRIO 2
#define CFG_CPRIO 0
#define CFG_STG0 180u
#define CFG_RATE1 90u
#define CFG_SLOPE 0.105
#define CFG_PGAP 30u
#define CFG_XS 18u
#define CFG_XD 0u
#define CFG_HF 0.0
#define CFG_PERM 0x71543260u
#define CFG_AFF 1
#define CFG_ADV 0u
#define CFG_RATE0 87u
#define CFG_P 169u
#define CFG_L0 80u
#define CFG_KS 2
#define CFG_PMAX 192u
#define CFG_PEND 180u
#define CFG_GEND (33u * 8u)
#define CFG_PSM 0
#define CFG_WIN 8u
#define CFG_F2 0
constexpr int NRANK = 8;
constexpr int NBC = CFG_NBC;
constexpr int NH = CFG_NH;
constexpr int NBI = 32 - NBC - NH;
constexpr int RC = 16 * NBC, RH = 512 - 16 * NH;
constexpr int BX = CFG_BX;
constexpr int NHR = NH + 2;
constexpr int HB = 2 * NHR, NI = HB + NBI;
constexpr int NA = NHR + 16 - NH / 2, NT = NA * 32;
constexpr int NC = NBC + NH;
constexpr u32 NX = NA - NHR, NPART = (NX + 32u) * 8u;
constexpr int NWG = 256;
constexpr int NPUSH = CFG_NPUSH;
constexpr int NRED = CFG_NRED;
constexpr u32 NPS = NPUSH * CFG_PW / 4;
constexpr int NGX = (NWG - NPUSH - NRED) / 8;
static_assert(NRED == 16 && NPUSH == 16 && NC <= 16 && NH == 2 && NI % 2 == 0, "reducer shares");
static_assert((NPUSH + NRED) % 8 == 0, "roles must keep the XCD round-robin aligned");
constexpr int NGEMM = NGX * 8;
constexpr int NTI = (NI - 4) * 16 + 2 * BX, NTC = NC * 16;
static_assert(NTI <= 24 * 16, "reducer shares");
constexpr int QMAX = 60;
constexpr int ITMAX = NT + 3 * (QMAX + 32 * NHR);
constexpr int WGI = 5;

constexpr u32 OFF_CTR = 0;
constexpr u32 OFF_TMO = 8192;
constexpr u32 OFF_IF = 16384;
constexpr u32 OFF_YF = 65536;
constexpr u32 FS = 704u * 512u;
constexpr u32 YT = 4096u, YO = NT * 2u * YT + CFG_YPAD, YP = 4u * YO;
#define CFG_OFFY (1u << 20)
constexpr u32 OFF_Y = CFG_OFFY;
constexpr u32 OFF_IN = OFF_Y + 2u * YP;
constexpr u32 WS_USED = OFF_IN + (12u << 20);
constexpr u32 IC = 32768u, IS = 384u * 4096u;
constexpr u64 DEADLINE = 50000000ull;
constexpr int AUX_SYS = 17;
constexpr u32 SRD3 = 0x00020000;

struct Params {
  const char* x;
  const char* w;
  char* me;
  u32 q, perm;
  char* out;
  char* ws[NRANK];
  union {
    u32 wg[3 * NWG];
    struct { u32 used[3 * NGEMM]; unsigned short rt[NTI + NTC]; };
  };
};
__host__ __device__ __forceinline__ u32 send_of(u32 s) {
  const u32 v = (s > CFG_XS + CFG_XD ? s : CFG_XS + CFG_XD) - CFG_XD, t = s < v ? s : v, u = t < 2u * NX ? t : 2u * NX;
  const u32 cx = u >> 1, cw = s + 16u - cx;
  return (u & 1u) ? cx : cw;
}
__host__ __device__ __forceinline__ u32 pace(u32 g) {
  return CFG_STG0 + g * CFG_RATE1 / 16u;
}

#define WAIT_VM0() do { asm volatile("" ::: "memory"); __builtin_amdgcn_s_waitcnt(0x0F70); asm volatile("" ::: "memory"); } while (0)
#define WAIT_LGKM0() do { asm volatile("" ::: "memory"); __builtin_amdgcn_s_waitcnt(0xC07F); asm volatile("" ::: "memory"); } while (0)
__device__ __forceinline__ u32 rfl(u32 v) { return __builtin_amdgcn_readfirstlane(v); }
__device__ __forceinline__ __amdgpu_buffer_rsrc_t mkrs(const void* p, u32 bytes) {
  return __builtin_amdgcn_make_buffer_rsrc((void*)p, (short)0, (int)bytes, SRD3);
}
__device__ __forceinline__ void vrows(u32 perm, u32 h, u32 o, u32 b, u32& r0, u32& r1) {
  const u32 t = h + (NBC - HB), u = t - (32 - NH), rh = ((32 - NH) + (h >> 1)) & 31u, rn = min(t, u);
  const bool hb = h < (u32)HB;
  const u32 row = hb ? rh : rn, px = hb ? h & 1u : 0u;
  const u32 pm = perm >> (((u >> 31) ^ 1u ^ px) * 16u), oo = (o - b * px) * 4u;
  r0 = ((pm >> (oo & 12u)) & 7u) << 19 | row << 14;
  r1 = ((pm >> ((oo + 4u) & 12u)) & 7u) << 19 | row << 14;
}

__device__ __forceinline__ u32 role_of(u32 bid) {
  return CFG_AFF == 0 || bid >= 32u ? bid : (((bid & 4u) != 0u) == (CFG_AFF == 1) ? 0u : 16u) + (bid >> 3) * 4u + (bid & 3u);
}

__device__ __forceinline__ bool wait_words(const u32* word, u32* tmo, u32 seq) {
  const u64 t0 = __builtin_amdgcn_s_memrealtime();
  for (u32 it = 0;; ++it) {
    const u32 v = __hip_atomic_load(word, __ATOMIC_RELAXED, __HIP_MEMORY_SCOPE_SYSTEM);
    if (__builtin_amdgcn_ballot_w64(v < seq) == 0) return true;
    if ((it & 7u) == 7u) {
      const u32 t = __hip_atomic_load(tmo, __ATOMIC_RELAXED, __HIP_MEMORY_SCOPE_AGENT);
      if (rfl(t) == seq || __builtin_amdgcn_s_memrealtime() - t0 > DEADLINE) {
        __hip_atomic_store(tmo, seq, __ATOMIC_RELAXED, __HIP_MEMORY_SCOPE_AGENT);
        return false;
      }
    }
    __builtin_amdgcn_s_sleep(2);
  }
}

__device__ __forceinline__ float bflo(u32 v) { return __builtin_bit_cast(float, v << 16); }
__device__ __forceinline__ float bfhi(u32 v) { return __builtin_bit_cast(float, v & 0xffff0000u); }
__device__ __forceinline__ u32 pkbf(float a, float b) {
  f32x2 f = {a, b};
  return __builtin_bit_cast(u32, __builtin_convertvector(f, bf16x2));
}

constexpr u32 PM = 0x87878787u;
__device__ __forceinline__ u32 bfi(u32 m, u32 x, u32 y) { return __builtin_amdgcn_bitop3_b32(m, x, y, 0xca); }
__device__ __forceinline__ void enc8(u32 d0, u32 d1, u32 d2, u32 d3, u32 K, u32& a0, u32& a1, u32& b, u32& c) {
  a0 = __builtin_amdgcn_perm(d1, d0, 0x06040200u);
  a1 = __builtin_amdgcn_perm(d3, d2, 0x06040200u);
  const u32 h0 = __builtin_amdgcn_perm(d1, d0, 0x07050301u) - K;
  const u32 h1 = __builtin_amdgcn_perm(d3, d2, 0x07050301u) - K;
  const u32 r = __builtin_amdgcn_alignbit(h1, h1, 28u);
  b = bfi(PM, h0, r);
  c = bfi(PM, r, h0);
}
__device__ __forceinline__ void dec_hi(u32 b, u32 c, u32 K, u32& h0, u32& h1) {
  const u32 r = bfi(PM, c, b);
  h0 = bfi(PM, b, c) + K;
  h1 = __builtin_amdgcn_alignbit(r, r, 4u) + K;
}
__device__ __forceinline__ u32 wave_max(float mx) {
  u32 m = __builtin_bit_cast(u32, mx);
  m = max(m, (u32)__builtin_amdgcn_update_dpp(0, (int)m, 0x128, 0xf, 0xf, false));
  m = max(m, (u32)__builtin_amdgcn_update_dpp(0, (int)m, 0x124, 0xf, 0xf, false));
  m = max(m, (u32)__builtin_amdgcn_update_dpp(0, (int)m, 0x122, 0xf, 0xf, false));
  m = max(m, (u32)__builtin_amdgcn_update_dpp(0, (int)m, 0x121, 0xf, 0xf, false));
  const u32 m0 = max((u32)__builtin_amdgcn_readlane((int)m, 0), (u32)__builtin_amdgcn_readlane((int)m, 16));
  const u32 m1 = max((u32)__builtin_amdgcn_readlane((int)m, 32), (u32)__builtin_amdgcn_readlane((int)m, 48));
  return max(m0, m1) & 0x7fffffffu;
}
__device__ __forceinline__ u32 block_k(u32 m) { return (max(m, 7u << 24) - (7u << 24)) >> 24; }
__device__ __forceinline__ u32 group_bits(u64 m) {
  m |= m >> 4; m |= m >> 2; m |= m >> 1;
  const u32 lo = (u32)m & 0x01010101u, hi = (u32)(m >> 32) & 0x01010101u;
  return ((lo * 0x01020408u) >> 24) | (((hi * 0x01020408u) >> 24) << 4);
}
__device__ __forceinline__ float amax4(float m, f32x4 v) {
  m = fmaxf(fmaxf(m, fabsf(v[0])), fabsf(v[1]));
  return fmaxf(fmaxf(m, fabsf(v[2])), fabsf(v[3]));
}
constexpr u32 EM = 0x44444444u;
constexpr u32 Y_B = 2048, Y_C = 2560, Y_M = 2816, Y_X = 3072;
__device__ __forceinline__ u32 bfn(u32 m, u32 x, u32 y) { return __builtin_amdgcn_bitop3_b32(m, x, y, 0x3a); }
__device__ __forceinline__ u32 tag_of(u32 seq) { return ((seq + 1u) << 30 | (seq + 1u) << 18) & 0x80100000u; }
__device__ __forceinline__ void lds_or(char AS3* a, u32 v) {
  (void)__hip_atomic_fetch_or((u32 AS3*)a, v, __ATOMIC_RELAXED, __HIP_MEMORY_SCOPE_WORKGROUP);
}
template <int NB>
__device__ __forceinline__ void stage(const f32x4 (&v)[NB][4], float mx, u32 seq, __amdgpu_buffer_rsrc_t rs, u32 s0, u32 ds, u32 n0,
                                      u32 dn, u32 lane, char AS3* fl, u32 dl) {
  constexpr int S = (NB + 1) / 2;
  const u32 k = block_k(max(wave_max(mx), CFG_OVER) - CFG_OVER), K = k * 0x01010101u;
  u32 b[S][4], m[S], fw = tag_of(seq);
#pragma unroll
  for (int i = 0; i < NB; ++i) {
    u32x4 a;
    u32x2 x;
#pragma unroll
    for (int u = 0; u < 2; ++u) {
      u32 a0, a1, cc;
      enc8(pkbf(v[i][2 * u][0], v[i][2 * u][1]), pkbf(v[i][2 * u][2], v[i][2 * u][3]), pkbf(v[i][2 * u + 1][0], v[i][2 * u + 1][1]),
           pkbf(v[i][2 * u + 1][2], v[i][2 * u + 1][3]), K, a0, a1, b[i >> 1][2 * (i & 1) + u], cc);
      a[2 * u] = a0; a[2 * u + 1] = a1; x[u] = cc;
    }
    __builtin_amdgcn_raw_buffer_store_b128(a, rs, lane * 16u + s0 + i * ds + (n0 + i * dn) * 1024u, 0, AUX_SYS);
    u32 hd = seq << 16 | k;
    const u64 mc = __builtin_amdgcn_ballot_w64((x[0] | x[1]) != 0u);
    if (mc) {
      __builtin_amdgcn_raw_buffer_store_b64(x, rs, lane * 8u + Y_X, s0 + i * ds + (n0 + i * dn) * 512u, AUX_SYS);
      hd |= group_bits(mc) << 7;
    }
    fw = lane == i * 8u + 7u - (n0 + i * dn) * 7u ? hd : fw;
  }
  if (NB == 1) { b[0][2] = b[0][0]; b[0][3] = b[0][1]; }
  const bool up = lane >= 32u;
  const u32 n = n0 + (up ? dn : 0u), l5 = lane & 31u, vb = up ? (NB > 1 ? ds : 0x80000000u) : 0u, l = l5 + n * 32u;
#pragma unroll
  for (int s = 0; s < S; ++s) {
    const u32x2 p = __builtin_amdgcn_permlane32_swap(b[s][0], b[s][2], false, false);
    const u32x2 r = __builtin_amdgcn_permlane32_swap(b[s][1], b[s][3], false, false);
    m[s] = ~bfi(EM << 1, r[1] << 1, bfi(EM >> 1, r[0] >> 1, bfi(EM >> 2, p[0] >> 2, p[1])));
    __builtin_amdgcn_raw_buffer_store_b64(u32x2{bfi(EM, p[1] << 2, p[0]), bfi(EM, p[1] << 1, r[0])}, rs, vb + l * 8u + Y_B, s0 + 2 * s * ds, AUX_SYS);
    __builtin_amdgcn_raw_buffer_store_b32(bfi(EM, p[1] >> 1, r[1]), rs, vb + l * 4u + Y_C, s0 + 2 * s * ds, AUX_SYS);
    __builtin_amdgcn_raw_buffer_store_b32(m[s], rs, vb + l * 4u + Y_M, s0 + 2 * s * ds, AUX_SYS);
  }
  __builtin_amdgcn_sched_barrier(0);
  if (lane < NB * 8u) *(volatile u32 AS3*)(fl + (lane >> 3) * dl + (lane & 7u) * 4u) = fw;
  const u32 below = (1u << l5) - 1u;
  char AS3* f = fl + (up ? dl : 0u) + n * 4u;
  if (NB == 1 && up) m[0] = 0u;
  u32 cnt[S], mm[S];
#pragma unroll
  for (int s = 0; s < S; ++s) { cnt[s] = m[s] & ~below & 1u ? 32u : 0u; mm[s] = m[s]; }
  u32 sh = lane & 32u;
  asm("" : "+v"(sh));
  u64 mk[S];
#pragma unroll
  for (int s = 0; s < S; ++s) mk[s] = __builtin_amdgcn_uicmp(mm[s], 0u, 33);
  do {
    u32 ad[S], pl[S];
#pragma unroll
    for (int s = 0; s < S; ++s) {
      const u32 mw = (u32)(mk[s] >> sh), pos = __builtin_popcount(mw & below) + cnt[s];
      u32 bit, lo;
      asm("v_mul_u32_u24 %0, 0x2ab, %1" : "=v"(bit) : "v"(pos));
      asm("v_ffbl_b32 %0, %1" : "=v"(lo) : "v"(mm[s]));
      asm("v_add_lshl_u32 %0, %1, %2, %3" : "=v"(pl[s]) : "v"(lo), "v"(l5 << 5), "v"(bit >> 6));
      ad[s] = bit >> 11;
      asm("" : "+v"(ad[s]));
      mk[s] &= __builtin_amdgcn_uicmp(pos, 21u, 36);
      cnt[s] += __builtin_popcount(mw);
      mm[s] &= mm[s] - 1u;
    }
#pragma unroll
    for (int s = 0; s < S; ++s) {
      if (__builtin_amdgcn_inverse_ballot_w64(mk[s])) lds_or(f + 2 * s * dl + ad[s] * 4u, pl[s]);
      mk[s] = __builtin_amdgcn_uicmp(mm[s], 0u, 33);
    }
  } while ((mk[0] | mk[S - 1]) != 0u);
#pragma unroll
  for (int s = 0; s < S; ++s)
    if (l5 == 0u && cnt[s] > 21u) lds_or(f + 2 * s * dl + 28u - n * 32u, 0x8000u);
}

__device__ __forceinline__ bool wait_flags(const u64* word, u32* tmo, u32 seq, u32x2& fl) {
  const u64 t0 = __builtin_amdgcn_s_memrealtime();
  for (u32 it = 0;; ++it) {
    const u64 v = __hip_atomic_load(word, __ATOMIC_RELAXED, __HIP_MEMORY_SCOPE_SYSTEM);
    fl[0] = (u32)v; fl[1] = (u32)(v >> 32);
    if (__builtin_amdgcn_ballot_w64(fl[0] < seq) == 0) return true;
    if ((it & 7u) == 7u) {
      const u32 t = __hip_atomic_load(tmo, __ATOMIC_RELAXED, __HIP_MEMORY_SCOPE_AGENT);
      if (rfl(t) == seq || __builtin_amdgcn_s_memrealtime() - t0 > DEADLINE) {
        __hip_atomic_store(tmo, seq, __ATOMIC_RELAXED, __HIP_MEMORY_SCOPE_AGENT);
        return false;
      }
    }
    __builtin_amdgcn_s_sleep(2);
  }
}

constexpr u32 L_RING = 256, L_WAVE = 2048;
__device__ __forceinline__ void scout(const Params& p, u32 q, char* me, u32 seq, u32 type, u32 tbase, u32 nent,
                                      u32 lane, char AS3* lds, u32 from) {
  u32* tmo = (u32*)(me + OFF_TMO);
  const __amdgpu_buffer_rsrc_t rme = mkrs(me, WS_USED);
  const u64 t0 = __builtin_amdgcn_s_memrealtime();
  u32 vo[3], id[3];
  u64 todo[3];
#pragma unroll
  for (u32 k = 0; k < 3; ++k) {
    const u32 ent = k * 8u + (lane >> 3);
    const u32 e = p.rt[tbase + (ent < nent ? ent : 0u) * 16u];
    const u32 t = (e & 1023u) + ((e >> 10) & lane & 1u), hf = (e & 1024u) ? type : lane & 1u;
    vo[k] = t * 512u + ((lane >> 1) & (t >= (u32)NT ? 1u : 3u)) * 128u + hf * 64u + 28u;
    id[k] = 0x80000000u | (t * 2u + hf);
    todo[k] = __builtin_amdgcn_ballot_w64(ent < nent) & 0x0303030303030303ull;
  }
  const u32 fbase = rfl(OFF_YF + (seq & 1u) * FS), fb2 = rfl(type ? 0u : fbase);
  u32 cnt = 0;
  u32x2 f[2][3];
  u64 seen[2][3];
  u32 gv = type ? OFF_IF + (seq & 1u) * 12288u + send_of(CFG_GATE + (lane >> 5)) * 256u + (lane & 31u) * 8u : 0x80000000u;
  u64 free0 = type ? __builtin_amdgcn_ballot_w64((id[0] & 2047u) >= 2u * NT) : ~0ull, free1 = type ? 0ull : ~0ull;
  if (type) while ((int)((u32)__builtin_amdgcn_s_memrealtime() - from) < 0) __builtin_amdgcn_s_sleep(16);
  auto poll = [&](u32x2 (&r)[3], u64 (&sn)[3]) {
    u32 open = 0;
#pragma unroll
    for (u32 k = 0; k < 3; ++k) {
      u64 m = sn[k] = open <= CFG_WIN ? todo[k] : 0ull;
      open += (u32)__builtin_popcountll(todo[k]);
      m |= m << 2; m |= m << 4;
      r[k] = __builtin_amdgcn_raw_buffer_load_b64(rme, __builtin_amdgcn_inverse_ballot_w64(m) ? vo[k] : k == 2 ? gv : 0x80000000u,
                                                  k == 2 ? fb2 : fbase, AUX_SYS);
    }
  };
  auto post = [&](u32 k, u64 nw, u32 word) {
    const u32 pos = __builtin_amdgcn_mbcnt_hi((u32)(nw >> 32), __builtin_amdgcn_mbcnt_lo((u32)nw, cnt));
    if ((nw >> lane) & 1u) *(volatile u32 AS3*)(lds + L_RING + pos * 4u) = word;
    cnt += (u32)__builtin_popcountll(nw);
    todo[k] &= ~nw;
  };
#define CFG_SPRIO 3
  __builtin_amdgcn_s_setprio(CFG_SPRIO);
  poll(f[0], seen[0]);
  u32 tp = (u32)t0;
  for (u32 it = 0;; ++it) {
#pragma unroll
    for (u32 u = 0; u < 2; ++u) {
      u32 now;
      while ((now = (u32)__builtin_amdgcn_s_memrealtime()) - tp < CFG_PGAP) __builtin_amdgcn_s_sleep(1);
      tp = now;
      poll(f[u ^ 1u], seen[u ^ 1u]);
#pragma unroll
      for (u32 k = 0; k < 3; ++k) {
        u64 m = __builtin_amdgcn_ballot_w64((int)((f[u][k][0] - (seq << 16)) | (f[u][k][1] - (seq << 16))) >= 0);
        m &= m >> 2; m &= m >> 4; m &= todo[k] & seen[u][k] & (k ? free1 : free0);
        if (m) post(k, m, id[k]);
      }
      if (__builtin_amdgcn_ballot_w64(f[u][2][0] < seq) == 0) {
        free0 = free1 = ~0ull;
        gv = 0x80000000u;
      }
    }
    if (cnt == 2u * nent) return;
    if ((it & 63u) == 63u) {
      const u32 t = __hip_atomic_load(tmo, __ATOMIC_RELAXED, __HIP_MEMORY_SCOPE_AGENT);
      if (rfl(t) == seq || __builtin_amdgcn_s_memrealtime() - t0 > DEADLINE) {
        __hip_atomic_store(tmo, seq, __ATOMIC_RELAXED, __HIP_MEMORY_SCOPE_AGENT);
#pragma unroll
        for (u32 k = 0; k < 3; ++k) post(k, todo[k], id[k] | 0x40000000u);
        return;
      }
    }
  }
}

__device__ __forceinline__ void reduce_units(const Params& p, u32 q, char* me, u32 seq, u32 type, u32 count, u32 lane,
                                             char AS3* lds, char AS3* ldw) {
  __amdgpu_buffer_rsrc_t ro = mkrs(p.out, 512u * 8192u);
  const char* src[4];
#pragma unroll
  for (u32 j = 0; j < 4; ++j) src[j] = p.ws[(type * 4u + ((q + 1u + j) & 3u)) & 7u];
  const __amdgpu_buffer_rsrc_t rme = mkrs(me, WS_USED);
  const u32 ybase = OFF_Y + (seq & 1u) * YP;
  const u64 t0 = __builtin_amdgcn_s_memrealtime();
  const u32 g = lane >> 4;
  const u32 vo = (lane & 15u) * 8192u + (16u * (g & 1u) + 8u * (g >> 1)) * 2u;
  const u32 vl = lane * 16u, gb = 7u + (lane >> 3), vx = lane * 8u + Y_X;
  const u32 vf0 = ((q + 1u + g) & 3u) * 128u + (lane & 15u) * 4u, vf1 = g < 2u ? g * 128u + (lane & 15u) * 4u : 0x80000000u;
  const bool lst = ((lane + 1u) & 14u) != 8u;
  char AS3* sc = ldw + (lane & 8u) * 64u + g * 4u;
  for (;;) {
    u32 n = 0;
    if (lane == 0) n = __hip_atomic_fetch_add((u32 AS3*)lds, 1u, __ATOMIC_RELAXED, __HIP_MEMORY_SCOPE_WORKGROUP);
    n = rfl(n);
    if (n >= count) break;
    u32 e;
#define CFG_RCK 255u
    for (u32 it = 0; (e = rfl(*(volatile const u32 AS3*)(lds + L_RING + n * 4u))) == 0u; ++it) {
      if ((it & CFG_RCK) == CFG_RCK && __builtin_amdgcn_s_memrealtime() - t0 > 3u * DEADLINE) return;
#define CFG_RSL 1
      __builtin_amdgcn_s_sleep(CFG_RSL);
    }
    bool ok = !(e & 0x40000000u);
    const u32 t = (e >> 1) & 1023u, hf = e & 1u;
    const u32 fo = OFF_YF + (seq & 1u) * FS + t * 512u + hf * 64u, vf = t >= (u32)NT ? vf1 : vf0;
    const u32 h = (t >> 5) * 2u + hf, b = t & 31u;
    __amdgpu_buffer_rsrc_t rp[4];
    u32 so[4];
#pragma unroll
    for (u32 j = 0; j < 4; ++j) { rp[j] = mkrs(src[j], WS_USED); so[j] = ybase + (h * 32u + b) * YT + q * YO; }
    u32 orow = type == 0 ? RC + 16u * (h - HB) : 16u * (h - NI), zk = 0u;
    if (t >= (u32)NT) {
      const u32 jh = h - 2u * NA, r = (b - 1u) & 3u;
      rp[0] = mkrs(r == 0 ? src[0] : r == 1 ? src[1] : r == 2 ? src[2] : src[3], WS_USED);
      so[0] = ybase + (jh * 64u + 32u + b) * YT + ((q + b) & 3u) * YO;
      rp[1] = rme;
      so[1] = ybase + (jh * 64u + b) * YT + q * YO;
      rp[2] = rp[3] = mkrs(me, 0u);
      zk = 4u;
      orow = (RH + 16u * jh) & 511u;
    }
    const bool fin = h >= (u32)HB;
    const u32 oo = orow * 8192u + b * 256u;
    if (ok) {
      u32x4 da[2][4];
      u32x2 db[2][4], dc[2][4];
      u32 vm[4], vc[2][4], hd[8];
      u32 hv = __builtin_amdgcn_raw_buffer_load_b32(rme, vf, fo, AUX_SYS);
      __builtin_amdgcn_sched_barrier(0);
#pragma unroll
      for (int j = 0; j < 4; ++j) {
        da[0][j] = __builtin_amdgcn_raw_buffer_load_b128(rp[j], vl, so[j], AUX_SYS);
        db[0][j] = __builtin_amdgcn_raw_buffer_load_b64(rp[j], lane * 8u + Y_B, so[j], AUX_SYS);
      }
#pragma unroll
      for (int j = 0; j < 4; ++j) db[1][j][1] = __builtin_amdgcn_raw_buffer_load_b32(rp[j], lane * 4u + Y_C, so[j], AUX_SYS);
      *(volatile u32x4 AS3*)(ldw + vl) = u32x4{0u, 0u, 0u, 0u};
      __builtin_amdgcn_sched_barrier(0);
      auto torn = [&]() { return __builtin_amdgcn_ballot_w64(lst && (int)vf >= 0 && ((hv ^ tag_of(seq)) & 0x80100000u)) != 0u; };
      if (__builtin_expect(torn(), 0)) {
        do {
          hv = __builtin_amdgcn_raw_buffer_load_b32(rme, vf, fo, AUX_SYS);
          ok = !torn();
        } while (!ok && __builtin_amdgcn_s_memrealtime() - t0 < DEADLINE);
      }
#pragma unroll
      for (int j = 0; j < 8; ++j) hd[j] = (u32)__builtin_amdgcn_readlane((int)hv, (j >> 1) * 16 + 7 + (j & 1));
      if (lst) {
#pragma unroll
        for (u32 i = 0; i < 32u; i += 10u + i / 10u) {
          const u32 pl = (hv >> i) & 1023u;
          lds_or(sc + ((hv >> (i + 5u)) & 31u) * 16u, min(pl, 1u) << (pl & 31u));
        }
      }
#pragma unroll
      for (int j = 0; j < 4; ++j) {
        vm[j] = ((lane < 32u ? hd[2 * j] : hd[2 * j + 1]) & 0x8000u) ? lane * 4u + Y_M : 0x80000000u;
        vc[0][j] = ((hd[2 * j] >> gb) & 1u) ? vx : 0x80000000u;
        vc[1][j] = ((hd[2 * j + 1] >> gb) & 1u) ? vx + 512u : 0x80000000u;
        asm volatile("" : "+v"(vm[j]), "+v"(vc[0][j]), "+v"(vc[1][j]));
      }
      __builtin_amdgcn_sched_barrier(0);
#pragma unroll
      for (int j = 0; j < 4; ++j) {
        db[1][j][0] = __builtin_amdgcn_raw_buffer_load_b32(rp[j], vm[j], so[j], AUX_SYS);
        dc[0][j] = __builtin_amdgcn_raw_buffer_load_b64(rp[j], vc[0][j], so[j], AUX_SYS);
      }
      const u32x4 ml = *(volatile const u32x4 AS3*)(ldw + vl);
      __builtin_amdgcn_sched_barrier(0);
      asm volatile("s_cmp_ge_u32 %0, %1\n\ts_cbranch_scc1 1\n\ts_waitcnt vmcnt(%2)" :: "s"(n + CFG_TAILN), "s"(count), "i"(CFG_HW) : "scc");
      __builtin_amdgcn_sched_barrier(0);
#pragma unroll
      for (int j = 0; j < 4; ++j) {
        da[1][j] = __builtin_amdgcn_raw_buffer_load_b128(rp[j], vl + 1024u, so[j], AUX_SYS);
        dc[1][j] = __builtin_amdgcn_raw_buffer_load_b64(rp[j], vc[1][j], so[j], AUX_SYS);
      }
      __builtin_amdgcn_sched_barrier(0);
#pragma unroll
      for (int j = 0; j < 4; ++j) {
        const u32 b0 = db[0][j][0], b1 = db[0][j][1], m = db[1][j][0] | ml[j], b3 = db[1][j][1];
        const u32 b2 = bfi(EM << 1, b3 << 1, bfi(EM >> 1, b1 >> 1, b0 >> 2));
        const u32x2 p = __builtin_amdgcn_permlane32_swap(bfn(EM, m << 2, b0), bfn(EM, m, b2), false, false);
        const u32x2 r = __builtin_amdgcn_permlane32_swap(bfn(EM, m << 1, b1), bfn(EM, m >> 1, b3), false, false);
        db[0][j] = u32x2{p[0], r[0]};
        db[1][j] = u32x2{p[1], r[1]};
      }
      f32x4 c[2][4];
#pragma unroll
      for (int n = 0; n < 2; ++n) {
#pragma unroll
        for (int j = 0; j < 4; ++j)
#pragma unroll
          for (int i = 0; i < 2; ++i) {
            u32 h[2];
            dec_hi(db[n][j][i], dc[n][j][i], ((hd[2 * j + n] & 127u) - (j < 2 ? 0u : zk)) * 0x01010101u, h[0], h[1]);
#pragma unroll
            for (int u = 0; u < 2; ++u) {
              f32x4 v;
#pragma unroll
              for (int e = 0; e < 4; ++e)
                v[e] = __builtin_bit_cast(float, __builtin_amdgcn_perm(h[u], da[n][j][2 * i + u], 0x04000c0cu + e * 0x01010000u));
              c[n][2 * i + u] = j == 0 ? v : c[n][2 * i + u] + v;
            }
          }
        __builtin_amdgcn_sched_barrier(0);
      }
      if (fin) {
#pragma unroll
        for (int i = 0; i < 4; ++i) {
          u32 o[4];
#pragma unroll
          for (int u = 0; u < 4; ++u) {
            const f32x4 v = c[i >> 1][2 * (i & 1) + (u >> 1)];
            o[u] = pkbf(fmaxf(v[2 * (u & 1)], 0.f), fmaxf(v[2 * (u & 1) + 1], 0.f));
          }
          const u32x2 s0 = __builtin_amdgcn_permlane16_swap(o[0], o[2], false, false);
          const u32x2 s1 = __builtin_amdgcn_permlane16_swap(o[1], o[3], false, false);
          const u32x4 r = {s0.x, s1.x, s0.y, s1.y};
          __builtin_amdgcn_raw_buffer_store_b128(r, ro, vo + oo + i * 64u, 0, CFG_OUT_AUX);
        }
      } else {
        float mx = 0.f;
#pragma unroll
        for (int i = 0; i < 8; ++i) mx = amax4(mx, c[i >> 2][i & 3]);
        stage<2>(c, mx, seq, rme, so[0], 0u, 0u, 1u, lane, ldw + 1024u, 32u);
        const u32x4 fl = *(volatile const u32x4 AS3*)(ldw + 1024u + (lane & 3u) * 16u);
        const __amdgpu_buffer_rsrc_t rf = mkrs(p.ws[hf * 4u + ((q - b * hf) & 3u)], WS_USED);
        WAIT_VM0();
        __builtin_amdgcn_raw_buffer_store_b128(fl, rf, lane < 4u && ok ? vl : 0x80000000u,
                                               OFF_YF + (seq & 1u) * FS + (NT + (h >> 2) * 32u + b) * 512u + (hf ^ 1u) * 128u + (h & 2u) * 32u, AUX_SYS);
      }
    }
    if (!ok && fin) {
      const u32x4 nanv = {0x7fc07fc0u, 0x7fc07fc0u, 0x7fc07fc0u, 0x7fc07fc0u};
#pragma unroll
      for (int i = 0; i < 4; ++i) __builtin_amdgcn_raw_buffer_store_b128(nanv, ro, vo + oo + i * 64u, 0, CFG_OUT_AUX);
    }
  }
}

struct Frag { bf16x8 x[4]; bf16x8 w[4]; };

template <int STAGE, int KS>
__device__ __forceinline__ void read_frag(Frag& f, const char AS3* lds, u32 xa0, u32 xa1, const u32 (&wa)[4]) {
  const u32 xa = KS ? xa1 : xa0;
  f.x[0] = *(const bf16x8 AS3*)(lds + xa + STAGE * 32768);
#pragma unroll
  for (int nt = 0; nt < 4; ++nt) {
    const s16x4 lo = __builtin_amdgcn_ds_read_tr16_b64_v4i16((s16x4 AS3*)(lds + wa[nt] + STAGE * 32768 + KS * 4096));
    const s16x4 hi = __builtin_amdgcn_ds_read_tr16_b64_v4i16((s16x4 AS3*)(lds + wa[nt] + STAGE * 32768 + KS * 4096 + 512));
    const s16x8 v = __builtin_shufflevector(lo, hi, 0, 1, 2, 3, 4, 5, 6, 7);
    f.w[nt] = __builtin_bit_cast(bf16x8, v);
  }
#pragma unroll
  for (int mt = 1; mt < 4; ++mt)
    f.x[mt] = *(const bf16x8 AS3*)(lds + xa + STAGE * 32768 + mt * 2048);
}

#define SCHED() __builtin_amdgcn_sched_barrier(0)
#define BAR() __builtin_amdgcn_s_barrier()
#define FENCE() do { asm volatile("" ::: "memory"); SCHED(); } while (0)
template <bool V> struct Tag { static constexpr bool v = V; };
#define OWN Tag<false>{}
#define REM Tag<true>{}
constexpr u32 LDS_FAIL = 131072;
constexpr u32 LDS_CNT = LDS_FAIL + 16u, LDS_HD = LDS_FAIL + 512u;
__device__ __forceinline__ u32 failed(const char AS3* lds) {
  const u32x4 f = *(volatile const u32x4 AS3*)(lds + LDS_FAIL);
  return f[0] | f[1] | f[2] | f[3];
}

__global__ void __launch_bounds__(512) __attribute__((amdgpu_num_sgpr(106))) fused_gemm_rs(const Params p) {
  __shared__ __attribute__((aligned(1024))) char lds_raw[131072 + 1024];
  const u32 ao = max((int)(offsetof(Params, ws) + 32u + (blockIdx.x & 3u) * 8u), (int)(offsetof(Params, wg) + (blockIdx.x - (NPUSH + NRED)) * 12u));
  asm volatile("s_load_dword s101, %0, 0x0\n\ts_load_dword s101, %0, %1\n\ts_load_dword s101, %0, %1 offset:0x8\n\ts_memrealtime s[100:101]"
               :: "s"(__builtin_amdgcn_kernarg_segment_ptr()), "s"(ao) : "s100", "s101");
  SCHED();
  const u32 tid = threadIdx.x, lane = tid & 63u;
  const u32 wave = rfl(tid >> 6);
  const u32 bid = role_of(blockIdx.x);
  const u32 slot = (bid - (NPUSH + NRED)) >> 3;
  const char *px, *pw;
  char* me;
  u64 its;
  u32 t00, q, perm, wgw, *tmo;
  const u32 co = OFF_CTR + (bid * 8u + wave) * 4u;
  auto args = [&]() {
    u64x4 aa;
    u64x2 ab;
    asm volatile("s_load_dwordx8 %0, %3, 0x0\n\ts_load_dwordx4 %1, %3, %4\n\ts_waitcnt lgkmcnt(0)\n\ts_mov_b32 %2, s100"
                 : "=&s"(aa), "=&s"(ab), "=&s"(t00) : "s"(__builtin_amdgcn_kernarg_segment_ptr()), "s"(ao));
    px = (const char*)aa[0];
    pw = (const char*)aa[1];
    me = (char*)(char AS1*)aa[2];
    q = (u32)aa[3] & 3u;
    perm = (u32)(aa[3] >> 32);
    its = ab[0];
    wgw = (u32)ab[1];
    tmo = (u32*)(me + OFF_TMO);
  };
  auto ctr_load = [&]() { return __builtin_amdgcn_raw_buffer_load_b32(mkrs(me, WS_USED), 0, co, AUX_SYS); };
  auto ctr_store = [&](u32 v) { __builtin_amdgcn_raw_buffer_store_b32(v, mkrs(me, WS_USED), 0, co, AUX_SYS); };

  const u32 vX0 = (lane >> 3) * 1024u + (((lane & 7u) ^ (lane >> 4)) * 16u), vX1 = (vX0 ^ 64u) + 8192u;
  const u32 vW0 = (lane >> 3) * 8192u + ((((lane & 7u) >> 1) ^ ((lane >> 4) & 1u)) * 32u) + (lane & 1u) * 16u;
  const u32 vW1 = (vW0 ^ 64u) + 65536u;
  const u32 vl = lane * 16u, gb = 8u + (lane >> 3);

  if (bid < (u32)(NPUSH + NRED)) {
    const bool cross = bid < (u32)NPUSH;
    const bool push = cross && wave < CFG_PW;
    ((volatile u32 AS3*)lds_raw)[tid & 127u] = 0u;
    u32 seq;
    if (push) {
      const u32 dq = bid & 3u, ps = CFG_PSM ? wave * 4u + (bid >> 2) : (bid & 12u) + wave;
      u32 b0, b1, sh, mk, v0, v1;
      auto plan = [&](u32 gi) {
        const u32 rq = (gi >> 1) & 3u, c = send_of(gi >> 3), t = 2u * c + NBC + (rq >> 1), u = t - (32 - NH);
        const bool isx = c < 16u;
        const u32 x0 = min(t, u) << 14 | dq << 8, w0 = (128u * dq + 32u * rq) * 8192u + (c - 16u) * 256u, w1 = w0 + 16u * 8192u;
        b0 = isx ? x0 : w0; b1 = isx ? x0 : w1; v0 = isx ? vX0 : vW0; v1 = isx ? vX1 : vW1;
        sh = (u >> 31) * 16u + (rq & 1u) * 8u; mk = isx ? 7u << 19 : 0u;
      };
      auto fetch = [&](u32x4 (&r)[2][4], u32 size = 4u << 20) {
        const u32 pm = perm >> sh, s0 = (pm << 19 & mk) | b0, s1 = (pm << 15 & mk) | b1;
        const __amdgpu_buffer_rsrc_t rs = mkrs(mk ? px : pw, size);
#pragma unroll
        for (u32 i = 0; i < 8; ++i)
          r[i >> 2][i & 3u] = __builtin_amdgcn_raw_buffer_load_b128(rs, ((i & 1u) ? v1 : v0) + (i >> 2) * 128u, (i & 2u) ? s1 : s0, 0);
      };
      auto encode = [&](const u32x4 (&v)[2][4], u32x4 (&o)[2][4], u32& k) {
        k = block_k(wave_max(amax4(amax4(0.f, __builtin_bit_cast(f32x4, v[0][0])), __builtin_bit_cast(f32x4, v[0][0] << 16))));
        const u32 K = k * 0x01010101u;
#pragma unroll
        for (int u = 0; u < 2; ++u) {
#pragma unroll
          for (int i = 0; i < 4; ++i) {
            u32 a0, a1, bb, cc;
            enc8(v[u][i][0], v[u][i][1], v[u][i][2], v[u][i][3], K, a0, a1, bb, cc);
            o[u][i >> 1][2 * (i & 1)] = a0; o[u][i >> 1][2 * (i & 1) + 1] = a1; o[u][2][i] = bb; o[u][3][i] = cc;
          }
          asm volatile("" : "+v"(o[u][0]), "+v"(o[u][1]), "+v"(o[u][2]), "+v"(o[u][3]));
        }
      };
      plan(2u * ps);
      asm volatile("" : "+s"(b0), "+s"(b1), "+s"(sh), "+s"(mk), "+v"(v0), "+v"(v1));
      args();
      const __amdgpu_buffer_rsrc_t rd = mkrs((const char*)its, WS_USED);
      const u32 tstart = t00;
      u32x4 v[2][4], o[2][4];
      u32x4 vf[2][4];
      fetch(vf);
      FENCE();
      const u32 seq0 = ctr_load();
      FENCE();
      plan(2u * (ps + NPS));
      if (CFG_F2 == 0) fetch(v);
      else asm volatile("" : "+s"(b0), "+s"(b1), "+s"(sh), "+s"(mk), "+v"(v0), "+v"(v1));
      FENCE();
      WAIT_LGKM0();
      BAR();
      FENCE();
      if (CFG_F2 == 1) fetch(v);
      FENCE();
      u32 k;
      encode(vf, o, k);
      FENCE();
      if (CFG_F2 == 2) fetch(v);
      FENCE();
      seq = rfl(seq0 + 1u);
      const u32 par = seq & 1u;
      const u32 fvo = OFF_IF + par * 12288u + (lane == 0 ? 0u : 0x80000000u);
      const u32 late = max((u32)__builtin_amdgcn_s_memrealtime() - tstart, CFG_ADV) - CFG_ADV;
      u32 dep = max(pace(2u * ps), late + 2u * ps * CFG_RATE0 / 16u);
      u32 gi = 2u * ps;
#pragma unroll 1
      do {
        const u32 idx = send_of(gi >> 3) * 32u + q * 8u + (gi & 7u);
        u32 dvo = vl + OFF_IN + par * (6u << 20) + (idx >> 5) * IC + q * IS + (gi & 7u) * 4096u;
        u32 dvo1 = dvo + 4096u;
        asm volatile("" : "+v"(dvo), "+v"(dvo1));
        u32 now;
        while ((now = (u32)__builtin_amdgcn_s_memrealtime() - tstart) < dep) __builtin_amdgcn_s_sleep(1);
        FENCE();
#pragma unroll
        for (int i = 0; i < 3; ++i) __builtin_amdgcn_raw_buffer_store_b128(o[0][i], rd, dvo + i * 1024u, 0, AUX_SYS);
#pragma unroll
        for (int i = 0; i < 3; ++i) __builtin_amdgcn_raw_buffer_store_b128(o[1][i], rd, dvo1 + i * 1024u, 0, AUX_SYS);
        FENCE();
        u32x4 fv = {seq, k, seq, k};
#pragma unroll
        for (int u = 0; u < 2; ++u) {
          fv[2 * u + 1] |= group_bits(__builtin_amdgcn_ballot_w64((o[u][3][0] | o[u][3][1] | o[u][3][2] | o[u][3][3]) != 0u)) << 8;
          __builtin_amdgcn_raw_buffer_store_b128(o[u][3], rd, ((fv[2 * u + 1] >> gb) & 1u) ? (u ? dvo1 : dvo) + 3072u : 0x80000000u, 0, AUX_SYS);
        }
        asm volatile("" : "+v"(fv));
        FENCE();
        encode(v, o, k);
        FENCE();
        plan(gi + 4u * NPS < NPART ? gi + 4u * NPS : gi);
        fetch(v, gi + 4u * NPS < NPART ? 4u << 20 : 0u);
        FENCE();
        __builtin_amdgcn_s_waitcnt(0x0F78);
        FENCE();
        __builtin_amdgcn_raw_buffer_store_b128(fv, rd, fvo + idx * 8u, 0, AUX_SYS);
        FENCE();
        const u32 lat = (u32)__builtin_amdgcn_s_memrealtime() - tstart - now;
        dep = (gi == 2u * ps ? dep : max(dep, now - 8u)) + min(CFG_P + ((max(lat, CFG_L0) - CFG_L0) >> CFG_KS), gi < CFG_GEND ? CFG_PMAX : CFG_PEND);
        gi += 2u * NPS;
      } while (gi < NPART);
      FENCE();
      __builtin_amdgcn_s_waitcnt(0x0F71);
      FENCE();
    } else {
      args();
      const u32 seq0 = ctr_load();
      FENCE();
      BAR();
      FENCE();
      seq = rfl(seq0 + 1u);
    }
    args();
    ctr_store(seq);
    if (7u - wave > (u32)(cross ? CFG_CW : CFG_RW)) return;
    const u32 nent = cross ? NC : (NTI + 15u + NPUSH - bid) / 16u;
    if (wave == 7u) {
      scout(p, q, me, seq, cross, cross ? NTI + bid : bid - NPUSH, nent, lane, (char AS3*)lds_raw, t00 + CFG_TA + bid * CFG_TAS);
      return;
    }
    reduce_units(p, q, me, seq, cross, 2u * nent, lane, (char AS3*)lds_raw, (char AS3*)lds_raw + (wave + 1u) * L_WAVE);
    return;
  }

  const char AS3* lds = (const char AS3*)lds_raw;
  auto item_of = [&](u32 i) { return (u32)(its >> (12u * i)) & 4095u; };
  const u32 g = lane >> 4, l15 = lane & 15u;
#define GEMM_ARGS() \
  args(); \
  const u32 nq = wgw & 7u, ntile = (wgw >> 3) & 7u, nh = (wgw >> 6) & 3u, nht = (wgw >> 8) & 1u; \
  const __amdgpu_buffer_rsrc_t rme = mkrs(me, WS_USED)

  if (wave >= 4u) {
    __builtin_amdgcn_s_setprio(CFG_LPRIO);
    const u32 lw = wave - 4u;
    *(volatile u32 AS3*)(lds + LDS_FAIL + lw * 4u + (lane & 1u) * 16u) = 0u;
    GEMM_ARGS();
    u32x4 AS3* img = (u32x4 AS3*)(lds + lw * 4096u + vl);
    const u32 nit = nq + ntile;
    u32 sx0, sx1, sw, rx, rw, ox, ow, vf, dk, rk, nrec;
    auto setup = [&](u32 i) {
      const u32 e = item_of(i);
      nrec = i < nit ? 4u << 20 : 0u;
      const u32 ea = (e >> 5) & 31u, eb = e & 31u, hm = (e >> 11) & 1u, hn = (e >> 10) & 1u;
      const bool sm = i < nq;
      const u32 l0 = lw & 1u, l1 = lw >> 1, ras = hm * 2u + l0, rks = (e >> 9) & 6u;
      const u32 ra = sm ? ras : lw, ca = sm ? l1 : 0u, rb = sm ? lw : l0, cb = sm ? hn : l1;
      vrows(perm, ea * 2u + (ra >> 1), (ra & 1u) * 2u, eb, sx0, sx1);
      sx0 += ca * 128u; sx1 += ca * 128u;
      sw = rb * (32u * 8192u) + eb * 256u + cb * 128u;
      dk = sm ? 2u : 1u;
      rk = sm ? rks : slot * 3u;
      rx = ((ea - NHR) & 15u) * 32u + ra * 2u + ca;
      rw = (16u + eb) * 32u + rb * 2u + cb;
      const u32 o = lane & dk & 1u;
      vf = OFF_IF + (((lane >> 1) & 3u) * 8u + ((lane & 8u) ? rw + o * 4u : rx + o)) * 8u;
    };
    u32x4 B[4][8];
    u32x2 fl;
    auto ld = [&](auto rem, u32 i, u32 kk) -> u32x4 {
      const u32 m = i >> 2;
      if (!rem.v) {
        const u32 so = m ? sw + kk * (64u * 8192u) + (i & 2u) * 65536u : ((i & 2u) ? sx1 : sx0) + kk * 128u;
        return __builtin_amdgcn_raw_buffer_load_b128(mkrs(m ? pw : px, nrec), (i & 1u) ? (m ? vW1 : vX1) : (m ? vW0 : vX0), so, 0);
      }
      const u32 so = (m ? ow + (kk & 1u) * 16384u : ox + (kk & 1u) * 4096u) + (kk >> 1) * IS;
      u32 vo = vl + (i & 3u) * 1024u;
      if ((i & 3u) == 3u) {
        const u32 hd = (u32)__builtin_amdgcn_readlane((int)fl[1], 8u * m + kk);
        vo = ((hd >> gb) & 1u) ? vo : 0x80000000u;
      }
      return __builtin_amdgcn_raw_buffer_load_b128(rme, vo, so, CFG_REM_AUX);
    };
    auto put_own = [&](u32x4 (&r)[8], u32 j, auto rem, u32 kl) {
#pragma unroll
      for (u32 i = 0; i < 8; ++i) {
        img[j * 2048u + (i >> 2) * 1024u + (i & 3u) * 64u] = r[i];
        FENCE();
        r[i] = ld(rem, i, kl);
        FENCE();
      }
    };
    auto put_rem = [&](u32x4 (&r)[8], u32 j, u32 kk, auto rem, u32 kl) {
#pragma unroll
      for (u32 m = 0; m < 2; ++m) {
        const u32 K = ((u32)__builtin_amdgcn_readlane((int)fl[1], 8u * m + kk) & 127u) * 0x01010101u;
#pragma unroll
        for (u32 n = 0; n < 2; ++n) {
          u32 h[4];
          dec_hi(r[4 * m + 2][2 * n], r[4 * m + 3][2 * n], K, h[0], h[1]);
          dec_hi(r[4 * m + 2][2 * n + 1], r[4 * m + 3][2 * n + 1], K, h[2], h[3]);
          asm volatile("" : "+v"(h[0]), "+v"(h[1]), "+v"(h[2]), "+v"(h[3]));
#pragma unroll
          for (u32 i = 0; i < 2; ++i) {
            if (n) { FENCE(); r[4 * m + 2 + i] = ld(rem, 4 * m + 2 + i, kl); FENCE(); }
            const u32 l0 = r[4 * m + n][2 * i], l1 = r[4 * m + n][2 * i + 1];
            img[j * 2048u + m * 1024u + (2 * n + i) * 64u] =
                u32x4{__builtin_amdgcn_perm(h[2 * i], l0, 0x05010400u), __builtin_amdgcn_perm(h[2 * i], l0, 0x07030602u),
                      __builtin_amdgcn_perm(h[2 * i + 1], l1, 0x05010400u), __builtin_amdgcn_perm(h[2 * i + 1], l1, 0x07030602u)};
            FENCE();
          }
          r[4 * m + n] = ld(rem, 4 * m + n, kl);
          FENCE();
        }
      }
    };
#define LBAR() do { FENCE(); WAIT_LGKM0(); BAR(); FENCE(); } while (0)
    setup(0u);
    if (__builtin_expect(wgw >= 512u, 0)) {
      const u32 dl = (wgw >> 9) * 4u;
      while ((u32)__builtin_amdgcn_s_memrealtime() - t00 < dl) __builtin_amdgcn_s_sleep(4);
    }
#define CFG_SEQAT 8u
    u32 seq0;
#pragma unroll
    for (u32 i = 0; i < CFG_SEQAT; ++i) {
      const u32 n = i < 8u ? i ^ 4u : i & 7u;
      B[i >> 3][n] = ld(OWN, n, ((i >> 3) * dk + rk) & 7u);
      FENCE();
      if (i == 0) { seq0 = ctr_load(); FENCE(); }
    }
    const u32 seq = rfl(seq0 + 1u);
    ctr_store(seq);
    const u32 fo = (seq & 1u) * 12288u, io = OFF_IN + (seq & 1u) * (6u << 20);
    FENCE();
#pragma unroll
    for (u32 i = CFG_SEQAT; i < 32; ++i) { B[i >> 3][i & 7u] = ld(OWN, i & 7u, ((i >> 3) * dk + rk) & 7u); FENCE(); }
    auto item = [&](auto sm, u32 i) {
      constexpr u32 dp = sm.v ? 2u : 1u;
      if (!sm.v) {
#pragma unroll
        for (u32 j = 0; j < 4; ++j) {
          put_own(B[j], j, OWN, (j + 4u + rk) & 7u);
          LBAR();
        }
      }
#pragma unroll
      for (u32 j = 0; j < 3; ++j) {
        if (j == 0) fl = __builtin_amdgcn_raw_buffer_load_b64(rme, vf, fo, AUX_SYS);
#pragma unroll
        for (u32 n = 0; n < 8; ++n) img[j * 2048u + (n >> 2) * 1024u + (n & 3u) * 64u] = B[j][n];
        LBAR();
      }
      if (__builtin_amdgcn_ballot_w64(fl[0] < seq) != 0)
        if (!wait_flags((const u64*)(me + fo + vf), tmo, seq, fl)) *(volatile u32 AS3*)(lds + LDS_FAIL + lw * 4u) = 1u;
      ox = io + (rx >> 5) * IC + (rx & 7u) * 4096u; ow = io + (rw >> 5) * IC + (rw & 7u) * 4096u;
      FENCE();
#pragma unroll
      for (u32 n = 0; n < 16; ++n) {
        B[n >> 3][n & 7u] = ld(REM, n & 7u, ((n >> 3) * dp + rk) & 7u);
        FENCE();
        if (n & 1u) img[3u * 2048u + (n >> 3) * 1024u + ((n >> 1) & 3u) * 64u] = B[3][n >> 1];
        FENCE();
      }
      LBAR();
      const u32 rp = rk;
      auto late = [&](u32 j) {
#pragma unroll
        for (u32 n = 0; n < 8; ++n) { B[j][n] = ld(REM, n, (j * dp + rp) & 7u); FENCE(); }
      };
      if (!sm.v) {
#pragma unroll
        for (u32 j = 0; j < 4; ++j) {
          if (j < 2) late(j + 2u);
          put_rem(B[j], j, (j + rk) & 7u, REM, (j + 4u + rk) & 7u);
          LBAR();
        }
      }
#pragma unroll
      for (u32 j = 0; j < 4; ++j) {
        if (j < 2 && sm.v) late(j + 2u);
        if (j == 0) { setup(i + 1u); FENCE(); }
        put_rem(B[j], j, ((j + 4u) * dp + rp) & 7u, OWN, (j * dk + rk) & 7u);
        LBAR();
      }
    };
    u32 i = 0;
    if (nht) {
#pragma unroll
      for (u32 j = 0; j < 4; ++j) {
        put_own(B[j], j, OWN, (j + 4u + rk) & 7u);
        LBAR();
      }
    }
#pragma unroll 1
    for (; i < nh + nht; ++i) {
      setup(i + 1u);
      FENCE();
#pragma unroll
      for (u32 j = 0; j < 4; ++j) {
        put_own(B[j], j, OWN, (j * dk + rk) & 7u);
        LBAR();
      }
    }
#pragma unroll 1
    for (; i < nq; ++i) item(Tag<true>{}, i);
#pragma unroll 1
    for (; i < nit; ++i) item(Tag<false>{}, i);
    BAR();
    return;
  }

  GEMM_ARGS();
  const u32 seq = rfl(ctr_load() + 1u);
  ctr_store(seq);
  __builtin_amdgcn_s_setprio(CFG_CPRIO);
  const u32 par = seq & 1u;
  const u32 wm = wave >> 1, wn = wave & 1u;
  char* wsi = p.ws[g] + OFF_YF + q * 128u + l15 * 8u;
  char* wsc = p.ws[4u + g] + OFF_YF + q * 128u + l15 * 8u;
  char AS3* lhd = (char AS3*)lds_raw + LDS_HD + wave * 32u;
  const u32 xa0 = (64u * wm + l15) * 128u + ((g ^ (l15 >> 1)) * 16u);
  const u32 xa1 = xa0 ^ 64u;
  u32 wa[4];
  {
    const u32 qq = l15 >> 2, pp = lane & 3u, f = (qq >> 1) | ((g & 1u) << 1);
#pragma unroll
    for (u32 nt = 0; nt < 4; ++nt) wa[nt] = 16384u + wn * 8192u + (8u * g + qq) * 128u + ((nt ^ f) * 32u) + 8u * pp;
  }
  if (nq) {
    const u32 xq0 = (16u * wave + l15) * 128u + ((g ^ (l15 >> 1)) * 16u), xq1 = xq0 ^ 64u;
    u32 wq[4];
#pragma unroll
    for (u32 nt = 0; nt < 4; ++nt) wq[nt] = wa[nt] - wn * 8192u;
#pragma unroll 1
    for (u32 i = 0; i < nq; ++i) {
      const u32 e = item_of(i);
      f32x4 acc1[1][4];
      f32x4 (&acc)[4] = acc1[0];
#pragma unroll
      for (int j = 0; j < 4; ++j) acc[j] = f32x4{0.f, 0.f, 0.f, 0.f};
#pragma unroll 1
      for (u32 half = i < nh; half < 2; ++half) {
#pragma unroll
        for (int kb = 0; kb < 4; ++kb) {
          SCHED();
          BAR();
          SCHED();
          bf16x8 xf[4], wf[4][4];
#pragma unroll
          for (int ks = 0; ks < 4; ++ks) {
            xf[ks] = *(const bf16x8 AS3*)(lds + ((ks & 1) ? xq1 : xq0) + (ks >> 1) * 8192 + kb * 32768);
#pragma unroll
            for (int nt = 0; nt < 4; ++nt) {
              const char AS3* ad = lds + wq[nt] + kb * 32768 + ks * 4096;
              const s16x4 lo = __builtin_amdgcn_ds_read_tr16_b64_v4i16((s16x4 AS3*)ad);
              const s16x4 hi = __builtin_amdgcn_ds_read_tr16_b64_v4i16((s16x4 AS3*)(ad + 512));
              wf[ks][nt] = __builtin_bit_cast(bf16x8, __builtin_shufflevector(lo, hi, 0, 1, 2, 3, 4, 5, 6, 7));
            }
          }
          SCHED();
#pragma unroll
          for (int ks = 0; ks < 4; ++ks)
#pragma unroll
            for (int nt = 0; nt < 4; ++nt)
              acc[nt] = __builtin_amdgcn_mfma_f32_16x16x32_bf16(wf[ks][nt], xf[ks], acc[nt], 0, 0, 0);
        }
        SCHED();
      }
      const u32 sub = (e >> 10) & 3u, h = ((e >> 5) & 31u) * 2u + (sub >> 1);
      const u32 own = ((h >= (u32)NI ? 4u : 0u) + wave) & 7u;
      const __amdgpu_buffer_rsrc_t rown = mkrs(p.ws[own], WS_USED);
      stage<1>(acc1, amax4(amax4(amax4(amax4(0.f, acc[0]), acc[1]), acc[2]), acc[3]), seq, rme,
               OFF_Y + par * YP + (h * 32u + (e & 31u)) * YT + wave * YO, 0u, sub & 1u, 0u, lane, lhd, 0u);
      const u32 fail = failed(lds);
      const u32x4 fl = *(volatile const u32x4 AS3*)(lhd + (lane & 1u) * 16u);
      WAIT_VM0();
      __builtin_amdgcn_raw_buffer_store_b128(fl, rown, (lane < 2u && fail == 0u) ? vl : 0x80000000u,
                                             OFF_YF + par * FS + (e & 1023u) * 512u + q * 128u + sub * 32u, AUX_SYS);
    }
  }
  BAR();
  Frag f0, f1;
  read_frag<0, 0>(f0, lds, xa0, xa1, wa);

#define MMA16(F) \
  _Pragma("unroll") for (int mt = 0; mt < 4; ++mt) \
    _Pragma("unroll") for (int nt = 0; nt < 4; ++nt) \
      acc[mt][nt] = __builtin_amdgcn_mfma_f32_16x16x32_bf16(F.w[nt], F.x[mt], acc[mt][nt], 0, 0, 0);
#define INTERLEAVE() \
  _Pragma("unroll") for (int i = 0; i < 6; ++i) { \
    __builtin_amdgcn_sched_group_barrier(0x008, 1, 0); \
    __builtin_amdgcn_sched_group_barrier(0x100, 2, 0); \
  } \
  __builtin_amdgcn_sched_group_barrier(0x008, 10, 0);
#define STEP(S) \
  SCHED(); \
  read_frag<(S), 1>(f1, lds, xa0, xa1, wa); \
  MMA16(f0) \
  INTERLEAVE() \
  SCHED(); \
  WAIT_LGKM0(); \
  BAR(); \
  SCHED(); \
  read_frag<((S) + 1) & 3, 0>(f0, lds, xa0, xa1, wa); \
  MMA16(f1) \
  INTERLEAVE()

#pragma unroll 1
  for (u32 tcount = 0; tcount < ntile; ++tcount) {
    const u32 e = item_of(nq + tcount) & 1023u;
    const u32 a = e >> 5, b = e & 31u;
    f32x4 acc[4][4];
#pragma unroll
    for (int i = 0; i < 4; ++i)
#pragma unroll
      for (int j = 0; j < 4; ++j) acc[i][j] = f32x4{0.f, 0.f, 0.f, 0.f};
#pragma unroll 1
    for (u32 it = tcount < nht ? 2u : 0u; it < 4; ++it) {
      STEP(0) STEP(1) STEP(2)
      STEP(3)
    }
    SCHED();
    const u32 h = 2u * a + wm;
    const u32 yo = OFF_Y + par * YP + (h * 32u + b) * YT;
    float mx = 0.f;
#pragma unroll
    for (int mt = 0; mt < 4; ++mt)
#pragma unroll
      for (int nt = 0; nt < 4; ++nt) mx = amax4(mx, acc[mt][nt]);
    stage<4>(acc, mx, seq, rme, yo, YO, wn, 0u, lane, lhd, 128u);
    const u32 fail = failed(lds);
    u64 AS1* fp = (u64 AS1*)((2u * a + ((lane >> 3) & 1u) >= (u32)NI ? wsc : wsi) + par * FS + e * 512u);
    asm volatile("" : "+v"(fp));
    WAIT_VM0();
    u32 c = 0;
    if (lane == 0) c = __builtin_amdgcn_atomic_inc32((u32 AS3*)(lds + LDS_CNT), 3u, __ATOMIC_RELAXED, "workgroup");
    const u64 hv = *(volatile const u64 AS3*)(lds + LDS_HD + lane * 8u);
    if (rfl(c) == 3u && fail == 0u)
      __hip_atomic_store(fp, hv, __ATOMIC_RELAXED, __HIP_MEMORY_SCOPE_SYSTEM);
  }
}

void build_tables(Params& p) {
  int pos[48] = {};
  for (u32 i = 0; i < NPART / 8u; ++i) pos[send_of(i)] = i;
  constexpr int E0 = NHR * 32;
  auto valid = [](int e) {
    const int a = e >> 5;
    return a == NI / 2 ? (e & 31) < 32 - BX : (a >= NH && a < NHR) || a > NA ? (e & 31) >= 32 - BX : true;
  };
  const int HS = (int)(CFG_HS) < 32 * NH ? (int)(CFG_HS) : 32 * NH, NB = 32 * NH + 2 * BX + 3 * HS;
  double arr[NT], rdy[NT];
  bool split[NT];
  for (int e = 0; e < NT; ++e) {
    const int px = pos[((e >> 5) - NHR) & 15], pw = pos[16 + (e & 31)];
    arr[e] = e < E0 ? 0.0 : CFG_A0 + CFG_AR * (px > pw ? px : pw);
    rdy[e] = e < E0 ? 0.0 : (arr[e] > CFG_RF ? arr[e] : CFG_RF) + CFG_RREM;
    split[e] = e < HS;
  }
  for (int k = 0, ns = 0; k < 100 && ns < QMAX; ++k) {
    const double d = CFG_DI + CFG_SLOPE * k;
    int have = 0, best = -1;
    for (int e = E0; e < NI / 2 * 32; ++e) {
      if (rdy[e] <= d) ++have;
      else if (!split[e] && arr[e] + CFG_QREM <= d && (best < 0 || arr[e] > arr[best])) best = e;
    }
    if (have > k || best < 0) continue;
    split[best] = true;
    rdy[best] = arr[best] + CFG_QREM;
    ++ns; --k;
  }
  int ord[NT], nv = 0;
  for (int e = 0; e < NT; ++e) {
    if (!valid(e)) continue;
    int i = nv++;
    for (; i > 0 && rdy[ord[i - 1]] > rdy[e]; --i) ord[i] = ord[i - 1];
    ord[i] = e;
  }
  struct Item { double arr, due; int e, k; };
  Item items[ITMAX];
  int cnt = 0, ni = 0, nc = 0;
  for (int n = 0; n < nv; ++n) {
    const int e = ord[n], a = e >> 5, b = e & 31;
    const double due = e < E0 ? CFG_DH + CFG_SLOPE * n : 2 * a < NI ? CFG_DI + 0.5 * CFG_SLOPE * ni : CFG_DC + 0.5 * CFG_SLC * nc;
    if (e >= E0) ni += 2 * a + 1 < NI ? 2 : 2 * a < NI;
    nc += 2 * a >= NI ? 2 : 2 * a + 1 >= NI;
    for (int sub = 0; sub < (split[e] ? 4 : 1); ++sub) {
      int i = cnt++;
      for (; i > NB && items[i - 1].due > due; --i) items[i] = items[i - 1];
      items[i] = Item{arr[e], due, e + (sub << 10), e < HS ? (sub & 1) + 2 * (b & 3) : e < E0 ? ((b >> 2) & 1) + 2 * (b & 3) : split[e] ? -1 : ((NX % 2 && a == NA - 1 ? (b < BX ? NI / 2 : b >> 2) : a) & 1) + 2 * (b & 3)};
    }
  }
  unsigned short list[NGEMM][8];
  int ch[NGEMM], cq[NGEMM], cr[NGEMM];
  double fr[NGEMM], st[NGEMM], pub[NT + E0 / 2];
  for (int i = 0; i < NGEMM; ++i) { ch[i] = cq[i] = cr[i] = 0; fr[i] = 0.0; st[i] = 0.0; }
  for (int i = 0; i < NT + E0 / 2; ++i) pub[i] = i < NT ? 0.0 : CFG_HF;
  for (int n = 0; n < cnt; ++n) {
    const Item& it = items[n];
    const bool brief = n < NB, small = n < 4 * HS || it.k < 0;
    const double rem = brief ? (small ? CFG_HREM : CFG_TREM) : small ? CFG_QREM : CFG_RREM;
    const double lim = it.due - (brief ? CFG_HSLACK : CFG_SLACK);
    int best = -1;
    double bt = 0.0, bk = 0.0;
    for (int w = it.k < 0 ? 0 : it.k; w < NGEMM; w += it.k < 0 ? 1 : 8) {
      if (cq[w] + cr[w] >= WGI || (small && cr[w]) || (brief && ch[w] >= (small ? 3 : 1))) continue;
      const bool first = cq[w] + cr[w] == 0;
      double t = fr[w] + (small ? (first ? CFG_QF : CFG_QOWN) : first ? CFG_RF : cr[w] ? CFG_ROWN : CFG_RQ);
      if (t < it.arr) t = it.arr;
      const double key = t + rem > lim ? t + rem : lim;
      if (best < 0 || key < bk - 1e-9 || (key < bk + 1e-9 && fr[w] > fr[best])) { best = w; bt = t; bk = key; }
    }
    if (cq[best] + cr[best] == 0) st[best] = bt - (small ? CFG_QF : CFG_RF);
    list[best][cq[best] + cr[best]] = (unsigned short)it.e;
    ch[best] += brief ? (small ? 1 : 4) : 0;
    if (small) ++cq[best]; else ++cr[best];
    fr[best] = bt + rem;
    if (fr[best] > pub[it.e & 1023]) pub[it.e & 1023] = fr[best];
    if (brief && small) fr[best] = bt + CFG_HFR;
  }
  ni = 0; nc = NTI;
  for (int e = 0; e < NT + E0 / 2; ++e) {
    const int a = e >> 5;
    if (!valid(e)) continue;
    if (2 * a + 1 == NI && (e & 1)) continue;
    if (2 * a + 1 == NI && pub[e + 1] > pub[e]) pub[e] = pub[e + 1];
    for (int type = 0; type < 2; ++type) {
      if (type ? 2 * a + 1 < NI : 2 * a >= NI) continue;
      int i = type ? nc++ : ni++;
      for (; i > (type ? NTI : 0) && pub[p.rt[i - 1] & 1023] > pub[e]; --i) p.rt[i] = p.rt[i - 1];
      p.rt[i] = (unsigned short)(e + (2 * a + 1 == NI ? 1024 : 0));
    }
  }
  for (int type = 0; type < 2; ++type) {
    unsigned short* rt = p.rt + type * NTI;
    const int rows = type ? NC : NTI / 16;
    for (int r = rows - 2; r >= 0; r -= 2)
      for (int i = 0; i < 8; ++i) { const unsigned short t = rt[r * 16 + i]; rt[r * 16 + i] = rt[r * 16 + 15 - i]; rt[r * 16 + 15 - i] = t; }
  }
  for (int w = 0; w < NGEMM; ++w) {
    u64 l = 0;
    for (int i = 0; i < cq[w] + cr[w]; ++i) l |= (u64)list[w][i] << (12 * i);
    p.used[3 * w] = (u32)l;
    p.used[3 * w + 1] = (u32)(l >> 32);
    p.used[3 * w + 2] = (u32)cq[w] | (u32)cr[w] << 3 | (u32)ch[w] << 6 | (u32)(st[w] * 25.0) << 9;
  }
}
static_assert(sizeof(Params) <= 4096, "kernel arguments");

}

extern "C" void kernel_launch(int rank, int nranks, ncclComm_t comm, hipStream_t stream,
                              void* const* d_in, const int* in_sizes, int n_in,
                              void* d_out, int out_size, const int* in_dtypes, int out_dtype,
                              void* const* d_out_all, void* const* d_ws_all, size_t ws_bytes) {
  (void)comm; (void)in_sizes; (void)n_in; (void)out_size; (void)in_dtypes; (void)out_dtype; (void)d_out_all;
  if (nranks != NRANK || (unsigned)rank >= (unsigned)NRANK || ws_bytes < WS_USED) {
    fprintf(stderr, "unsupported configuration\n");
    return;
  }
  Params p;
  p.x = (const char*)d_in[0];
  p.w = (const char*)d_in[1];
  p.out = (char*)d_out;
  const u32 place = (CFG_PERM >> (4 * rank)) & 7u;
  p.q = place & 3u;
  p.perm = place < 4u ? CFG_PERM : CFG_PERM >> 16 | CFG_PERM << 16;
  for (int r = 0; r < NRANK; ++r) p.ws[r] = (char*)d_ws_all[(p.perm >> (4 * r)) & 7u];
  p.me = (char*)d_ws_all[rank];
  build_tables(p);
  fused_gemm_rs<<<dim3(NWG), dim3(512), 0, stream>>>(p);
  HIP_CHECK(hipGetLastError());
}
```
